```python
import numpy as np

kernel_name = "gpu_gemm_a2a_m4096_k4096_n4096_f32_relu"


ABSMAX_RTOL = 0.001

COMPUTE_DTYPE = "float32"
EPILOGUE = "relu"
IN_QUANT = None
OUT_QUANT = None
IN_DTYPES = {"x": "float32", "w_mat": "float32"}
WORLD_SIZE = 8

def setup_inputs():

    x = np.random.randn(4096, 4096).astype(np.float32)
    w_mat = (np.random.randn(4096, 4096) / np.sqrt(4096)).astype(np.float32)
    return {"x": x, "w_mat": w_mat}

def _epilogue(y):
    return np.maximum(y, 0.0).astype(np.float32)

def reference(x, w_mat):
    y = (x @ w_mat).astype(np.float32)
    return _epilogue(y)
```

```cpp
#include <hip/hip_runtime.h>
#include <rccl/rccl.h>
#include <cstdio>

#define HIP_CHECK(expr) do { hipError_t _e = (expr); if (_e != hipSuccess) { \
    fprintf(stderr, "HIP error %s at %s:%d\n", hipGetErrorString(_e), __FILE__, __LINE__); \
    return; } } while (0)

namespace {

constexpr int NR = 8, MLOC = 512, KD = 4096, ND = 4096, NLOC = 512;
constexpr int NWG = 256, NTHR = 256;

#define CFG_MT 1
#define CFG_D1 3
#define CFG_DX 8
#define CFG_STAMP 0
constexpr int MT = CFG_MT, D1 = CFG_D1, DX = CFG_DX;
constexpr int NT = 8 / MT;
constexpr int SL = 8;
constexpr int SA = (256 - 16 * MT) / 16;
constexpr int SV = 32 - SL - SA;
static_assert(32 % DX == 0, "x window must divide the 32 k-steps");

constexpr unsigned WS_SEQ = 0;
constexpr unsigned WS_TMO = 1024;
constexpr unsigned WS_XFLAG = 2048;
constexpr unsigned WS_ARR = 4096;
constexpr unsigned ARR_STRIDE = 64;
constexpr unsigned WS_STAMP = 32768;
constexpr unsigned WS_X16 = 65536;
constexpr unsigned X16_BYTES = MLOC * KD * 2;
constexpr size_t WS_NEED = WS_X16 + X16_BYTES;
constexpr unsigned OUT_BYTES = (unsigned)NR * MLOC * NLOC * 4;
constexpr unsigned W_BYTES = (unsigned)KD * ND * 4;
constexpr unsigned X_BYTES = (unsigned)MLOC * KD * 4;
constexpr unsigned long long DEADLINE_TICKS = 50000000ull;

constexpr int LDS_W = 4 * SL * 4096;
constexpr int LDS_RED = 32768;
constexpr int LDS_BYTES = LDS_W + LDS_RED;
constexpr int NBUF = LDS_RED / (16384 * MT);

constexpr int AUX_SC1 = 16, AUX_SYS = 17;

typedef float f4 __attribute__((ext_vector_type(4)));
typedef unsigned u4 __attribute__((ext_vector_type(4)));
typedef unsigned u2 __attribute__((ext_vector_type(2)));
typedef _Float16 h8 __attribute__((ext_vector_type(8)));
typedef _Float16 h2 __attribute__((ext_vector_type(2)));
#define GAS __attribute__((address_space(1)))
#define LAS __attribute__((address_space(3)))
typedef GAS unsigned gu32;

struct Params {
    const float* x;
    const float* w;
    float* out[NR];
    char* ws[NR];
    int rank;
    int pad;
};

__device__ __forceinline__ unsigned pk(unsigned a, unsigned b) {
    h2 v = {(_Float16)__builtin_bit_cast(float, a), (_Float16)__builtin_bit_cast(float, b)};
    return __builtin_bit_cast(unsigned, v);
}
__device__ __forceinline__ unsigned long long now() { return __builtin_amdgcn_s_memrealtime(); }
#define SCHED() __builtin_amdgcn_sched_barrier(0)
#define STAMP(i) do { } while (0)
#define STAMPC(c, i) do { } while (0)

__global__ __launch_bounds__(NTHR, 1) void gemm_a2a(Params p) {
    extern __shared__ __attribute__((aligned(16))) char smem_[];
    LAS char* smem = (LAS char*)smem_;
    const int b = blockIdx.x;
    const int l = threadIdx.x & 63;
    const int w = __builtin_amdgcn_readfirstlane(threadIdx.x >> 6);
    const int xcd = b & 7, rg = (b >> 3) & 3, u = b >> 5;
    const int g = l >> 4, q = l & 15;
    char* const ws = p.ws[p.rank];
    STAMP(0);

    const __amdgpu_buffer_rsrc_t rsX = __builtin_amdgcn_make_buffer_rsrc((void*)p.x, 0, X_BYTES, 0x00020000);
    const __amdgpu_buffer_rsrc_t rsW = __builtin_amdgcn_make_buffer_rsrc((void*)p.w, 0, W_BYTES, 0x00020000);
    const __amdgpu_buffer_rsrc_t rsH = __builtin_amdgcn_make_buffer_rsrc((void*)(ws + WS_X16), 0, X16_BYTES, 0x00020000);
    const __amdgpu_buffer_rsrc_t rsO = __builtin_amdgcn_make_buffer_rsrc((void*)p.out[u], 0, OUT_BYTES, 0x00020000);

    u4 xs[8];
    {
        const int voff = (l >> 3) * (KD * 4) + (l & 7) * 16;
        const int soff = (b >> 3) * 16 * (KD * 4) + ((b & 7) * 512 + w * 128) * 4;
#pragma unroll
        for (int i = 0; i < 8; ++i)
            xs[i] = __builtin_amdgcn_raw_buffer_load_b128(rsX, voff, soff + (i & 1) * 8 * (KD * 4) + (i >> 1) * 128, 0);
    }
    const unsigned call = __hip_atomic_load((gu32*)(ws + WS_SEQ) + b, __ATOMIC_RELAXED, __HIP_MEMORY_SCOPE_AGENT) + 1u;
    SCHED();

    const int rot = (rg + u) & 3;
    auto kstep = [&](int s) { return 4 * ((s & ~3) | ((s + rot) & 3)) + w; };

    u4 st[D1][8];
    const int voffW = g * 8 * (ND * 4) + q * 16;
    const int colW = (u * 8 + xcd) * 64 * 4;
    auto issueW = [&](int s) {
        const int so = kstep(s) * 32 * (ND * 4) + colW;
#pragma unroll
        for (int j = 0; j < 8; ++j)
            st[s % D1][j] = __builtin_amdgcn_raw_buffer_load_b128(rsW, voffW, so + j * (ND * 4), 0);
    };
#pragma unroll
    for (int s = 0; s < D1; ++s) issueW(s);
    SCHED();

    {
        LAS char* sc = smem + LDS_W + w * 4096;
        const int woff = ((((l & 7) >> 1) * 16 + (l >> 3)) * 16) + (l & 1) * 8;
#pragma unroll
        for (int i = 0; i < 8; ++i) {
            u2 v = {pk(xs[i][0], xs[i][1]), pk(xs[i][2], xs[i][3])};
            *(LAS u2*)(sc + (i >> 1) * 1024 + (i & 1) * 128 + woff) = v;
        }
        __builtin_amdgcn_fence(__ATOMIC_ACQ_REL, "wavefront");
        const int so = ((b >> 3) * 128 + (b & 7) * 16 + w * 4) * 1024;
#pragma unroll
        for (int ks = 0; ks < 4; ++ks) {
            const u4 v = *(LAS u4*)(sc + ks * 1024 + l * 16);
            __builtin_amdgcn_raw_buffer_store_b128(v, rsH, l * 16, so + ks * 1024, AUX_SC1);
        }
    }
    SCHED();
    STAMP(1);

    u4 Ba[SA][4], Bv[SV][4];
    h8 xw[DX][MT];
    LAS char* wl = smem + w * (SL * 4096) + l * 16;
    const int xbase = rg * 8 * 128 * 1024;
    auto issueX = [&](int t, int s) {
#pragma unroll
        for (int m = 0; m < MT; ++m) {
            const int so = t < NT ? xbase + ((t * MT + m) * 128 + kstep(s)) * 1024 : (int)X16_BYTES;
            xw[s % DX][m] = __builtin_bit_cast(h8, __builtin_amdgcn_raw_buffer_load_b128(rsH, l * 16, so, AUX_SC1));
        }
    };
    gu32* const xflag = (gu32*)(ws + WS_XFLAG);
    gu32* const tmo = (gu32*)(ws + WS_TMO);
    unsigned xf = 0;
    bool bad = false;
    constexpr int SPOLL = 16;
    static_assert(SPOLL > D1 && SPOLL + D1 <= 32 - D1, "poll placement");
#pragma unroll
    for (int s = 0; s < 32; ++s) {
#pragma unroll
        for (int c = 0; c < 4; ++c) {
            const u4* r = st[s % D1];
            u4 f = {pk(r[0][c], r[1][c]), pk(r[2][c], r[3][c]), pk(r[4][c], r[5][c]), pk(r[6][c], r[7][c])};
            if (s < SL) *(LAS u4*)(wl + (s * 4 + c) * 1024) = f;
            else if (s < SL + SA) { u4 a; asm("; to acc" : "=a"(a) : "0"(f)); Ba[s - SL][c] = a; }
            else Bv[s - SL - SA][c] = f;
        }
        SCHED();
        if (s == D1) {
            asm volatile("s_waitcnt vmcnt(%0)" :: "i"(8 * (D1 - 1)) : "memory");
            __builtin_amdgcn_s_barrier();
            if (threadIdx.x == 0) __hip_atomic_store(xflag + b, call, __ATOMIC_RELAXED, __HIP_MEMORY_SCOPE_AGENT);
        }
        if (s == SPOLL + D1) {
            if (!__all(xf >= call)) {
                const unsigned long long t0 = now();
                while (true) {
                    xf = __hip_atomic_load(xflag + rg * 64 + l, __ATOMIC_RELAXED, __HIP_MEMORY_SCOPE_AGENT);
                    if (__all(xf >= call)) break;
                    if (__hip_atomic_load(tmo, __ATOMIC_RELAXED, __HIP_MEMORY_SCOPE_AGENT) == call || now() - t0 > DEADLINE_TICKS) {
                        __hip_atomic_store(tmo, call, __ATOMIC_RELAXED, __HIP_MEMORY_SCOPE_AGENT);
                        bad = true;
                        break;
                    }
                    __builtin_amdgcn_s_sleep(2);
                }
            }
            __builtin_amdgcn_fence(__ATOMIC_ACQUIRE, "wavefront");
        }
        if (s + D1 < 32) issueW(s + D1);
        if (s == SPOLL) xf = __hip_atomic_load(xflag + rg * 64 + l, __ATOMIC_RELAXED, __HIP_MEMORY_SCOPE_AGENT);
        if (s + D1 >= 32) {
#pragma unroll
            for (int z = 0; z < DX; ++z)
                if (z * D1 / DX == s + D1 - 32) issueX(0, z);
        }
        SCHED();
    }
    STAMP(2);

    const float poison = bad ? __builtin_nanf("") : 0.0f;
    const int voffO = (4 * g + w) * (NLOC * 4) + q * 16;
    const int baseO = (p.rank * MLOC + rg * 128) * (NLOC * 4) + xcd * 64 * 4;
    u4 Bl[4];
#pragma unroll
    for (int c = 0; c < 4; ++c) Bl[c] = *(LAS u4*)(wl + c * 1024);
#pragma unroll 1
    for (int t = 0; t < NT; ++t) {
        f4 acc[MT][4];
#pragma unroll
        for (int m = 0; m < MT; ++m)
#pragma unroll
            for (int c = 0; c < 4; ++c) acc[m][c] = f4{0.f, 0.f, 0.f, 0.f};
#pragma unroll
        for (int s = 0; s < 32; ++s) {
#pragma unroll
            for (int c = 0; c < 4; ++c) {
                const u4 f = s < SL ? Bl[c] : (s < SL + SA ? Ba[s < SL ? 0 : (s < SL + SA ? s - SL : 0)][c]
                                                          : Bv[s >= SL + SA ? s - SL - SA : 0][c]);
#pragma unroll
                for (int m = 0; m < MT; ++m)
                    acc[m][c] = __builtin_amdgcn_mfma_f32_16x16x32_f16(xw[s % DX][m], __builtin_bit_cast(h8, f), acc[m][c], 0, 0, 0);
                if (s + 1 < SL) Bl[c] = *(LAS u4*)(wl + ((s + 1) * 4 + c) * 1024);
                if (s == 28) Bl[c] = *(LAS u4*)(wl + c * 1024);
            }
            SCHED();
            if (s + DX < 32) issueX(t, s + DX);
            else issueX(t + 1, s + DX - 32);
            SCHED();
        }
        LAS char* red = smem + LDS_W + (t % NBUF) * (16384 * MT);
        if (NBUF == 1) __syncthreads();
#pragma unroll
        for (int m = 0; m < MT; ++m)
#pragma unroll
            for (int j = 0; j < 4; ++j)
                *(LAS f4*)(red + ((w * MT + m) * 4 + j) * 1024 + l * 16) =
                    f4{acc[m][0][j], acc[m][1][j], acc[m][2][j], acc[m][3][j]};
        __syncthreads();
#pragma unroll
        for (int m = 0; m < MT; ++m) {
            f4 v = *(LAS f4*)(red + ((0 * MT + m) * 4 + w) * 1024 + l * 16);
#pragma unroll
            for (int o = 1; o < 4; ++o) v += *(LAS f4*)(red + ((o * MT + m) * 4 + w) * 1024 + l * 16);
#pragma unroll
            for (int c = 0; c < 4; ++c) v[c] = (v[c] < 0.f ? 0.f : v[c]) + poison;
            __builtin_amdgcn_raw_buffer_store_b128(__builtin_bit_cast(u4, v), rsO, voffO,
                                                   baseO + (t * MT + m) * 16 * (NLOC * 4), AUX_SYS);
        }
        SCHED();
        STAMPC(t == 0, 3); STAMPC(t == 1, 4); STAMPC(t == NT / 2, 5); STAMPC(t == NT - 1, 6);
    }

    asm volatile("s_waitcnt vmcnt(0)" ::: "memory");
    __syncthreads();
    STAMP(11);
    if (threadIdx.x == 0) {
        if (u != p.rank)
            __hip_atomic_store((gu32*)(p.ws[u] + WS_ARR + (p.rank * 32 + rg * 8 + xcd) * ARR_STRIDE), call,
                               __ATOMIC_RELAXED, __HIP_MEMORY_SCOPE_SYSTEM);
        __hip_atomic_store((gu32*)(ws + WS_SEQ) + b, call, __ATOMIC_RELAXED, __HIP_MEMORY_SCOPE_AGENT);
    }
    if (u == p.rank && rg == 0 && xcd == 0) {
        gu32* f = (gu32*)(ws + WS_ARR + threadIdx.x * ARR_STRIDE);
        const bool skip = (int)(threadIdx.x >> 5) == p.rank;
        const unsigned long long t0 = now();
        while (true) {
            const bool ok = skip || __hip_atomic_load(f, __ATOMIC_RELAXED, __HIP_MEMORY_SCOPE_SYSTEM) >= call;
            if (__all(ok)) break;
            if (__hip_atomic_load(tmo, __ATOMIC_RELAXED, __HIP_MEMORY_SCOPE_AGENT) == call || now() - t0 > DEADLINE_TICKS) {
                __hip_atomic_store(tmo, call, __ATOMIC_RELAXED, __HIP_MEMORY_SCOPE_AGENT);
                p.out[u][(size_t)p.rank * MLOC * NLOC + threadIdx.x] = __builtin_nanf("");
                break;
            }
            __builtin_amdgcn_s_sleep(1);
        }
    }
}

}

extern "C" void kernel_launch(int rank, int nranks, ncclComm_t comm,
                              hipStream_t stream,
                              void* const* d_in, const int* in_sizes, int n_in,
                              void* d_out, int out_size,
                              const int* in_dtypes, int out_dtype,
                              void* const* d_out_all,
                              void* const* d_ws_all, size_t ws_bytes) {
    if (nranks != NR || (unsigned)rank >= (unsigned)NR || n_in != 2 || ws_bytes < WS_NEED ||
        in_sizes[0] != MLOC * KD || in_sizes[1] != KD * ND || out_size != NR * MLOC * NLOC) {
        fprintf(stderr, "kernel_launch: unexpected problem shape\n");
        return;
    }
    HIP_CHECK(hipFuncSetAttribute((const void*)gemm_a2a, hipFuncAttributeMaxDynamicSharedMemorySize, LDS_BYTES));
    Params p;
    p.x = (const float*)d_in[0];
    p.w = (const float*)d_in[1];
    for (int i = 0; i < NR; ++i) {
        p.out[i] = (float*)d_out_all[i];
        p.ws[i] = (char*)d_ws_all[i];
    }
    p.rank = rank;
    p.pad = 0;
    gemm_a2a<<<NWG, NTHR, LDS_BYTES, stream>>>(p);
    HIP_CHECK(hipGetLastError());
}
```

```cpp
#include <hip/hip_runtime.h>
#include <rccl/rccl.h>
#include <cstdio>
#include <utility>

#define HIP_CHECK(expr) do { hipError_t _e = (expr); if (_e != hipSuccess) { \
    fprintf(stderr, "HIP error %s at %s:%d\n", hipGetErrorString(_e), __FILE__, __LINE__); \
    return; } } while (0)

namespace {

constexpr int NR = 8, MLOC = 512, KD = 4096, ND = 4096, NLOC = 512;
constexpr int NWG = 256, NTHR = 256;

#define CFG_D1 2
#define CFG_DX 13
#define CFG_XS 31
#define CFG_MAP 2
#define CFG_G 4
#define CFG_GS 8
#define CFG_DELAY 0
#define CFG_DGRP 0
#define CFG_PHASE 1
#define CFG_NOSEND 0
#define CFG_NOXLOAD 0
#define CFG_NOMFMA 0
#define CFG_NOBAR 0
#define CFG_NOFIN 0
#define CFG_NOBL 0
#define CFG_NOLD 0
#define CFG_SWZ 0
#define CFG_OLO 0
#define CFG_OHI 1000
#define CFG_NOGRAB 0
#define CFG_STAMP 0
constexpr int D1 = CFG_D1, DX = CFG_DX;
constexpr int SL = 8, SA = 15, SV = 9;
static_assert(SL + SA + SV == 32 && DX >= 4 && DX <= 24, "k-step split");
constexpr int NPK = 4;
constexpr int HDR = 22;
constexpr int PK_WORDS = HDR + 512;
constexpr int PK_CH = 3;
constexpr unsigned SLOT = 2304;
static_assert((PK_WORDS + 3) * 4 <= SLOT && SLOT <= PK_CH * 1024, "slot size");

constexpr unsigned WS_SEQ = 0;
constexpr unsigned WS_TMO = 1024;
constexpr unsigned WS_XFLAG = 2048;
constexpr unsigned WS_STAMP = 32768;
constexpr unsigned WS_X16 = 65536;
constexpr unsigned X16_BYTES = MLOC * KD * 2;
constexpr unsigned WS_RECV = WS_X16 + X16_BYTES;
constexpr unsigned WAVE_SLOTS = NPK * SLOT;
constexpr unsigned RECV_BYTES = 2 * NR * 32 * 4 * WAVE_SLOTS;
constexpr size_t WS_NEED = (size_t)WS_RECV + RECV_BYTES;
constexpr unsigned OUT_BYTES = (unsigned)NR * MLOC * NLOC * 4;
constexpr unsigned W_BYTES = (unsigned)KD * ND * 4;
constexpr unsigned X_BYTES = (unsigned)MLOC * KD * 4;
constexpr unsigned long long DEADLINE_TICKS = 50000000ull;
constexpr unsigned TAG = 0x80000000u;

constexpr int LDS_W = 4 * SL * 4096;
constexpr int LDS_RED = LDS_W;
constexpr int LDS_STG = LDS_RED + 16384;
constexpr int STG_B = PK_CH * 1024 + 256;
constexpr int LDS_BYTES = LDS_STG + 4 * STG_B;
static_assert(LDS_BYTES <= 163840, "LDS");

constexpr int AUX_SC1 = 16, AUX_SYS = 17;

typedef float f4 __attribute__((ext_vector_type(4)));
typedef unsigned u4 __attribute__((ext_vector_type(4)));
typedef unsigned u2 __attribute__((ext_vector_type(2)));
typedef unsigned long long u64;
typedef _Float16 h8 __attribute__((ext_vector_type(8)));
typedef _Float16 h2 __attribute__((ext_vector_type(2)));
#define GAS __attribute__((address_space(1)))
#define LAS __attribute__((address_space(3)))
typedef GAS unsigned gu32;

struct Params {
    const float* x;
    const float* w;
    float* out;
    char* ws[NR];
    int rank;
    int pad;
};

template <int... I, class F>
__device__ __forceinline__ void sfor_(std::integer_sequence<int, I...>, F&& f) { (f(std::integral_constant<int, I>{}), ...); }
template <int N, class F>
__device__ __forceinline__ void sfor(F&& f) { sfor_(std::make_integer_sequence<int, N>{}, f); }

__device__ __forceinline__ unsigned pk(unsigned a, unsigned b) {
    h2 v = {(_Float16)__builtin_bit_cast(float, a), (_Float16)__builtin_bit_cast(float, b)};
    return __builtin_bit_cast(unsigned, v);
}
__device__ __forceinline__ u64 now() { return __builtin_amdgcn_s_memrealtime(); }
template <int K>
__device__ __forceinline__ unsigned scan_step(unsigned x) {
    constexpr int ctrl = K < 4 ? 0x110 + (1 << K) : (K == 4 ? 0x142 : 0x143);
    constexpr int rows = K < 4 ? 0xf : (K == 4 ? 0xa : 0xc);
    return x + (unsigned)__builtin_amdgcn_update_dpp(0, (int)x, ctrl, rows, 0xf, false);
}
#define SCHED() __builtin_amdgcn_sched_barrier(0)
#define LDS_BARRIER() do { asm volatile("s_waitcnt lgkmcnt(0)" ::: "memory"); __builtin_amdgcn_s_barrier(); \
                           asm volatile("" ::: "memory"); } while (0)
#define STAMP(i) do { } while (0)

constexpr bool in_lds(int s) { return s % 4 == 2; }
constexpr int lds_idx(int s) { return s / 4; }
constexpr int reg_idx(int s) { int n = 0; for (int i = 0; i < s; ++i) n += in_lds(i) ? 0 : 1; return n; }
constexpr int p1_slot(int i) {
    if (i < SL) return 2 + 4 * i;
    int n = i - SL;
    for (int s = 0; s < 32; ++s) if (!in_lds(s)) { if (n == 0) return s; --n; }
    return -1;
}

enum Op { PUT0, PUT1, PUT2, PUT3, BAR1, GET0, GET1, GET2, GET3, ADD0  , CMP0 = ADD0 + 12  , SEL0 = CMP0 + 4  ,
          OWN0 = SEL0 + 4, OWN1, NIB0, NIB1, NIB2, NIB3, SCN0  , TAG0 = SCN0 + 6  , TOT = TAG0 + 4, ADR0, ADR1, CNT,
          WR0  , HD0 = WR0 + 12, PAD0, PAD1, PAD2, RD0, RD1, RD2, ST0  ,
          BAR2 = ST0 + 6, NOPS };
struct Plan { int gap[NOPS]; };
constexpr int base_load(int G) {
    return (G % 4 == 0) + ((G / 4) % 4 == 3) + (G % 8 == 7);
}
constexpr Plan make_plan() {
    Plan p{};
    int used[128] = {};
    int G = 3;
    for (int k = 0; k < NOPS; ++k) {
        int cost = 1, lo = G;
        if (k <= PUT3 || k == BAR1) cost = 2;
        if (k == BAR1) lo = 17;
        if (k >= GET0 && k <= GET3) lo = G + 1;
        if (k == ADD0) lo = p.gap[GET3] + 10;
        if (k == SEL0) lo = p.gap[CMP0 + 3] + 1;
        if (k == SCN0) lo = p.gap[OWN1] + 2;
        if (k > SCN0 && k < SCN0 + 6) lo = p.gap[k - 1] + 2;
        if (k == TAG0) lo = p.gap[SCN0];
        if (k == TOT) lo = p.gap[SCN0 + 5] + 2;
        if (k == RD0) lo = G + 2;
        if (k == ST0) lo = p.gap[RD2] + 5;
        if (k >= ST0 && k < ST0 + 6 && (k - ST0) % 2 == 1) { lo = p.gap[k - 1] + 1; cost = 2; }
        if (k == BAR2) lo = G + 1;
        while (base_load(lo) + used[lo] + cost > 2) ++lo;
        used[lo] += cost;
        p.gap[k] = lo;
        G = lo;
    }
    return p;
}
constexpr Plan PLAN = make_plan();
static_assert(PLAN.gap[BAR2] < 124 && PLAN.gap[TAG0 + 3] <= PLAN.gap[TOT] && PLAN.gap[PUT3] < 12 && PLAN.gap[BAR1] < 28, "plan");
constexpr int op_at(int G, int n) {
    for (int k = 0; k < NOPS; ++k)
        if (PLAN.gap[k] == G && n-- == 0) return k;
    return -1;
}
constexpr int lds_ops_in(int a, int b, int h) {
    int n = 0;
    for (int k = 0; k < NOPS; ++k) {
        const bool lds = k <= PUT3 || (k >= GET0 && k <= GET3) || (k >= WR0 && k < WR0 + 12 && (k - WR0) % 3 == 1) ||
                         (h == 0 && (k == HD0 || k == PAD2 || (k >= RD0 && k <= RD2)));
        n += lds && PLAN.gap[k] > a && PLAN.gap[k] <= b;
    }
    return n;
}
constexpr int waitcnt_imm(int vm, int lgkm) { return (vm & 15) | (7 << 4) | ((lgkm & 15) << 8) | ((vm >> 4) << 14); }

__global__ __launch_bounds__(NTHR, 1) void gemm_a2a(Params p) {
    extern __shared__ __attribute__((aligned(16))) char smem_[];
    LAS char* smem = (LAS char*)smem_;
    const int b = blockIdx.x;
    const int l = threadIdx.x & 63;
    const int w = __builtin_amdgcn_readfirstlane(threadIdx.x >> 6);
    const int rg = (b >> 3) & 3;
    const int ci = CFG_MAP == 2 ? (b & 7) : (b >> 5);
    const int u = CFG_MAP == 2 ? (b >> 5) : (b & 7);
    const int g = l >> 4, q = l & 15;
    char* const ws = p.ws[p.rank];
    char* wsu = p.ws[u];
    float* outp = p.out;
    STAMP(0);

    const __amdgpu_buffer_rsrc_t rsX = __builtin_amdgcn_make_buffer_rsrc((void*)p.x, 0, X_BYTES, 0x00020000);
    const __amdgpu_buffer_rsrc_t rsW = __builtin_amdgcn_make_buffer_rsrc((void*)p.w, 0, W_BYTES, 0x00020000);
    const __amdgpu_buffer_rsrc_t rsH = __builtin_amdgcn_make_buffer_rsrc((void*)(ws + WS_X16), 0, X16_BYTES, 0x00020000);
    const int swz = (CFG_MAP == 2 ? u : ci) & CFG_SWZ;
    auto tileA = [&](int k) {
        return __builtin_amdgcn_make_buffer_rsrc((void*)(ws + WS_X16 + (rg * 8 + ((k ^ swz) & 7)) * (128 * 1024)), 0,
                                                 k < 8 ? 128 * 1024 : 0, 0x00020000);
    };

    u4 xs[8];
    {
        const int voff = (l >> 3) * (KD * 4) + (l & 7) * 16;
        const int soff = (b >> 3) * 16 * (KD * 4) + ((b & 7) * 512 + w * 128) * 4;
#pragma unroll
        for (int i = 0; i < 8; ++i)
            xs[i] = __builtin_amdgcn_raw_buffer_load_b128(rsX, voff, soff + (i & 1) * 8 * (KD * 4) + (i >> 1) * 128, 0);
    }
    const unsigned done = __hip_atomic_load((gu32*)(ws + WS_SEQ) + b, __ATOMIC_RELAXED, __HIP_MEMORY_SCOPE_AGENT);
    SCHED();

    const int line0 = (int)(((size_t)p.w >> 7) & 127) + (u * 8 + ci) * 2;
    const bool slowp = (line0 & 6) == 2 || (line0 & 126) == 62;
    const int gsz = (slowp ? CFG_GS : CFG_G);
    const int rot = rg * (gsz / 4);
    const int ph = (CFG_PHASE == 1 ? ((b >> 5) + (b & 7)) & 3 : (CFG_PHASE == 2 ? (b >> 5) & 3 : (CFG_PHASE == 3 ? (b >> 6) & 3 : 0))) * (gsz / 4);
    auto chunk = [&](int i) { const int a = (i - ph) & 31; return (((a & ~(gsz - 1)) | ((a + rot) & (gsz - 1))) + ph) & 31; };
    if (CFG_DELAY) {
        const int dg = CFG_DGRP == 0 ? (slowp ? 0 : 1) : (CFG_DGRP == 1 ? ((u + ci) & 1) : (CFG_DGRP == 2 ? ((u + ci) & 3) : u));
        const u64 t0 = now();
        while (now() - t0 < (u64)(dg * CFG_DELAY)) __builtin_amdgcn_s_sleep(1);
    }

    u4 st[D1][8];
    const int voffW = g * 8 * (ND * 4) + q * 16;
    const int colW = (u * 8 + ci) * 64 * 4 + w * 32 * (ND * 4);
    auto issueW = [&](int i) {
        const int so = chunk(i) * 128 * (ND * 4) + colW;
#pragma unroll
        for (int j = 0; j < 8; ++j)
            st[i % D1][j] = __builtin_amdgcn_raw_buffer_load_b128(rsW, voffW, so + j * (ND * 4), 0);
    };
#pragma unroll
    for (int i = 0; i < D1; ++i) issueW(i);
    SCHED();

    {
        LAS char* sc = smem + LDS_RED + w * 4096;
        const int woff = ((((l & 7) >> 1) * 16 + (l >> 3)) * 16) + (l & 1) * 8;
#pragma unroll
        for (int i = 0; i < 8; ++i) {
            u2 v = {pk(xs[i][0], xs[i][1]), pk(xs[i][2], xs[i][3])};
            *(LAS u2*)(sc + (i >> 1) * 1024 + (i & 1) * 128 + woff) = v;
        }
        __builtin_amdgcn_fence(__ATOMIC_ACQ_REL, "wavefront");
        const int so = ((b >> 3) * 128 + (b & 7) * 16 + w * 4) * 1024;
#pragma unroll
        for (int ks = 0; ks < 4; ++ks) {
            const u4 v = *(LAS u4*)(sc + ks * 1024 + l * 16);
            __builtin_amdgcn_raw_buffer_store_b128(v, rsH, l * 16, so + ks * 1024, AUX_SC1);
        }
    }
    const unsigned call = __builtin_amdgcn_readfirstlane(done) + 1u;
    asm volatile("" : "+s"(wsu), "+s"(outp));
    SCHED();
    STAMP(1);

    u4 Br[SA + SV][4];
    h8 xw[64 + DX];
    LAS char* wl = smem + w * (SL * 4096) + l * 16;
    int xo[32];
    sfor<32>([&](auto I) { xo[p1_slot(I)] = (chunk(I) * 4 + w) * 1024; });
    auto issueX = [&](__amdgpu_buffer_rsrc_t rs, int z) {
        xw[z] = __builtin_bit_cast(h8, __builtin_amdgcn_raw_buffer_load_b128(rs, CFG_NOXLOAD ? 128 * 1024 : l * 16, xo[z % 32], AUX_SC1));
    };
    gu32* const xflag = (gu32*)(ws + WS_XFLAG);
    gu32* const tmo = (gu32*)(ws + WS_TMO);
    unsigned xf = 0;
    bool bad = false;
    constexpr int SPOLL = 16;
    static_assert(SPOLL > D1 && SPOLL + D1 < CFG_XS, "poll placement");
    sfor<32>([&](auto I) {
        constexpr int i = I, s = p1_slot(i);
#pragma unroll
        for (int c = 0; c < 4; ++c) {
            const u4* r = st[i % D1];
            u4 f = {pk(r[0][c], r[1][c]), pk(r[2][c], r[3][c]), pk(r[4][c], r[5][c]), pk(r[6][c], r[7][c])};
            if constexpr (i == 31) f[0] = bad ? 0x7e007e00u : f[0];
            if constexpr (in_lds(s)) *(LAS u4*)(wl + (lds_idx(s) * 4 + c) * 1024) = f;
            else if constexpr (reg_idx(s) < SA) { u4 a; asm("; to acc" : "=a"(a) : "0"(f)); Br[reg_idx(s)][c] = a; }
            else Br[reg_idx(s)][c] = f;
        }
        SCHED();
        if constexpr (i == D1) {
            asm volatile("s_waitcnt vmcnt(%0)" :: "i"(8 * (D1 - 1)) : "memory");
            __builtin_amdgcn_s_barrier();
            if (threadIdx.x == 0) __hip_atomic_store(xflag + b, call, __ATOMIC_RELAXED, __HIP_MEMORY_SCOPE_AGENT);
        }
        if constexpr (i == SPOLL + D1) {
            if (!__all(xf >= call)) {
                const u64 t0 = now();
                while (true) {
                    xf = __hip_atomic_load(xflag + rg * 64 + l, __ATOMIC_RELAXED, __HIP_MEMORY_SCOPE_AGENT);
                    if (__all(xf >= call)) break;
                    if (__hip_atomic_load(tmo, __ATOMIC_RELAXED, __HIP_MEMORY_SCOPE_AGENT) == call || now() - t0 > DEADLINE_TICKS) {
                        __hip_atomic_store(tmo, call, __ATOMIC_RELAXED, __HIP_MEMORY_SCOPE_AGENT);
                        bad = true;
                        break;
                    }
                    __builtin_amdgcn_s_sleep(2);
                }
            }
            __builtin_amdgcn_fence(__ATOMIC_ACQUIRE, "wavefront");
        }
        if constexpr (i + D1 < 32) issueW(i + D1);
        if constexpr (i == SPOLL) xf = __hip_atomic_load(xflag + rg * 64 + l, __ATOMIC_RELAXED, __HIP_MEMORY_SCOPE_AGENT);
        if constexpr (i >= CFG_XS) {
#pragma unroll
            for (int z = 0; z < DX - 1; ++z)
                if (z * (32 - CFG_XS) / (DX - 1) == i - CFG_XS) { issueX(tileA(0), z); SCHED(); }
        }
        SCHED();
    });
    STAMP(2);

    const unsigned slot0 = ((((call & 1) * NR + p.rank) * 32 + rg * 8 + ci) * 4 + w) * WAVE_SLOTS;
    const __amdgpu_buffer_rsrc_t rsS =
        __builtin_amdgcn_make_buffer_rsrc((void*)(wsu + WS_RECV + slot0), 0, WAVE_SLOTS, 0x00020000);
    LAS char* const redw = smem + LDS_RED + w * 4096 + l * 16;
    LAS char* const redr = smem + LDS_RED + w * 1024 + l * 16;
    LAS char* const stg = smem + LDS_STG + w * STG_B;

    f4 acc[4], T[4], v;
    u4 Bl[4], sd[PK_CH];
    bool nz[4];
    unsigned cnt = 0, bits = 0, nib = 0, pos = 0, own = 0;
    const int hoff = (l / 3) * 4 + l % 3;
    *(LAS unsigned*)(stg + (l < HDR ? l : PK_CH * 256 + l) * 4) = TAG;
#pragma unroll
    for (int c = 0; c < 4; ++c) acc[c] = f4{0.f, 0.f, 0.f, 0.f};
#pragma unroll
    for (int j = 0; j < 4; ++j) T[j][0] = 0.f;

    auto grab = [&](int c) {
#pragma unroll
        for (int j = 0; j < 4; ++j) { float t; asm("; to vgpr" : "=v"(t) : "0"(acc[c][j])); T[j][c] = t; }
    };
    auto putred = [&](int j) { *(LAS f4*)(redw + j * 1024) = T[j]; };
    auto getred = [&](int o) { T[o] = *(LAS f4*)(redr + o * 4096); };
    auto addred = [&](int c) {
        float a = T[0][c] + T[1][c], e = T[2][c] + T[3][c];
        asm("" : "+v"(a));
        v[c] = a + e;
    };
    auto packA = [&](int c) {
        nz[c] = !(v[c] <= 0.f);
        nib = (c == 0 ? 0u : nib) | (nz[c] ? 1u << c : 0u);
    };
    auto packB = [&](int hh) {
        own = __builtin_popcount(nib);
        pos = own;
        bits = hh == 0 ? nib : bits | (nib << 4);
    };
    auto packC = [&]() {
        const unsigned tot = __builtin_amdgcn_readlane(pos, 63);
        pos = pos - own + (HDR + cnt);
        cnt += tot;
    };
    auto packD = [&](int c) {
        const float vc = v[c];
        *(LAS unsigned*)(stg + (nz[c] ? pos : PK_CH * 256 + l) * 4) = __float_as_uint(vc) | TAG;
        pos += nz[c] ? 1u : 0u;
    };
    auto sendA = [&]() {
        *(LAS unsigned char*)(stg + hoff) = (unsigned char)bits;
        *(LAS unsigned*)(stg + (l < 3 ? HDR + cnt + l : PK_CH * 256 + l) * 4) = TAG;
        __builtin_amdgcn_fence(__ATOMIC_ACQ_REL, "wavefront");
    };
    auto sendB = [&](int ch) { sd[ch] = *(LAS u4*)(stg + ch * 1024 + l * 16); };
    auto sendC = [&](int ch, unsigned n, int pkt) {
        if ((unsigned)(ch * 256 + l * 4) < n) __builtin_amdgcn_raw_buffer_store_b128(sd[ch], rsS, ch * 1024 + l * 16, pkt * SLOT, AUX_SYS);
    };

    const unsigned l16 = l * 16;

    int sso = -(int)SLOT;
    float t0[4] = {}, t1[4] = {};
    unsigned d4[4] = {}, tg[4] = {}, ownb = 0, nb0 = 0, nb1 = 0, adr = 0, sel = 0x7fff0000u, tot = 0;
    u64 mk[4] = {}, smk = 0;
    const unsigned dump = (unsigned)(size_t)(stg + (PK_CH * 256 + l) * 4);
    unsigned wbase = (unsigned)(size_t)stg;
    const unsigned padv = l < 3 ? l * 4u : 0x10000u;
    auto side = [&](auto KK, auto HH) {
        constexpr int k = KK, h = HH;
        if constexpr (k < CFG_OLO || k >= CFG_OHI) { }
        else if constexpr (k <= PUT3) putred(k - PUT0);
        else if constexpr (k == BAR1) {
            asm volatile("s_waitcnt lgkmcnt(4)" ::: "memory");
            if (!CFG_NOBAR) __builtin_amdgcn_s_barrier();
            asm volatile("" ::: "memory");
        }
        else if constexpr (k <= GET3) getred(k - GET0);
        else if constexpr (k < ADD0 + 4) asm volatile("v_add_f32 %0, %1, %2" : "=v"(t0[k - ADD0]) : "v"(T[0][k - ADD0]), "v"(T[1][k - ADD0]));
        else if constexpr (k < ADD0 + 8) asm volatile("v_add_f32 %0, %1, %2" : "=v"(t1[k - ADD0 - 4]) : "v"(T[2][k - ADD0 - 4]), "v"(T[3][k - ADD0 - 4]));
        else if constexpr (k < ADD0 + 12) asm volatile("v_add_f32 %0, %1, %2" : "=v"(t0[k - ADD0 - 8]) : "v"(t0[k - ADD0 - 8]), "v"(t1[k - ADD0 - 8]));
        else if constexpr (k < CMP0 + 4) asm volatile("v_cmp_nge_f32 %0, 0, %1" : "=s"(mk[k - CMP0]) : "v"(t0[k - CMP0]));
        else if constexpr (k < SEL0 + 4) asm volatile("v_cndmask_b32 %0, 0, 4, %1" : "=v"(d4[k - SEL0]) : "s"(mk[k - SEL0]));
        else if constexpr (k == OWN0) asm volatile("v_add3_u32 %0, %1, %2, %3" : "=v"(ownb) : "v"(d4[0]), "v"(d4[1]), "v"(d4[2]));
        else if constexpr (k == OWN1) asm volatile("v_add_u32 %0, %1, %2" : "=v"(ownb) : "v"(ownb), "v"(d4[3]));
        else if constexpr (k == NIB0) asm volatile("v_lshl_add_u32 %0, %1, 1, %2" : "=v"(nb0) : "v"(d4[1]), "v"(d4[0]));
        else if constexpr (k == NIB1) asm volatile("v_lshl_add_u32 %0, %1, 1, %2" : "=v"(nb1) : "v"(d4[3]), "v"(d4[2]));
        else if constexpr (k == NIB2) asm volatile("v_lshl_add_u32 %0, %1, 2, %2" : "=v"(nb0) : "v"(nb1), "v"(nb0));
        else if constexpr (k == NIB3) {
            if constexpr (h == 1) asm volatile("v_lshrrev_b32 %0, 2, %1" : "=v"(bits) : "v"(nb0));
            else asm volatile("v_lshl_or_b32 %0, %1, 2, %2" : "=v"(bits) : "v"(nb0), "v"(bits));
        }
        else if constexpr (k == SCN0) asm volatile("v_add_u32_dpp %0, %1, %1 row_shr:1 row_mask:0xf bank_mask:0xf bound_ctrl:1" : "=&v"(adr) : "v"(ownb));
        else if constexpr (k == SCN0 + 1) asm volatile("v_add_u32_dpp %0, %0, %0 row_shr:2 row_mask:0xf bank_mask:0xf bound_ctrl:1" : "+v"(adr));
        else if constexpr (k == SCN0 + 2) asm volatile("v_add_u32_dpp %0, %0, %0 row_shr:4 row_mask:0xf bank_mask:0xf bound_ctrl:1" : "+v"(adr));
        else if constexpr (k == SCN0 + 3) asm volatile("v_add_u32_dpp %0, %0, %0 row_shr:8 row_mask:0xf bank_mask:0xf bound_ctrl:1" : "+v"(adr));
        else if constexpr (k == SCN0 + 4) asm volatile("v_add_u32_dpp %0, %0, %0 row_bcast:15 row_mask:0xa bank_mask:0xf" : "+v"(adr));
        else if constexpr (k == SCN0 + 5) asm volatile("v_add_u32_dpp %0, %0, %0 row_bcast:31 row_mask:0xc bank_mask:0xf" : "+v"(adr));
        else if constexpr (k < TAG0 + 4) asm volatile("v_or_b32 %0, 0x80000000, %1" : "=v"(tg[k - TAG0]) : "v"(t0[k - TAG0]));
        else if constexpr (k == TOT) asm volatile("v_readlane_b32 %0, %1, 63" : "=s"(tot) : "v"(adr));
        else if constexpr (k == ADR0) asm volatile("v_sub_u32 %0, %1, %2" : "=v"(adr) : "v"(adr), "v"(ownb));
        else if constexpr (k == ADR1) asm volatile("v_add_u32 %0, %1, %2" : "=v"(adr) : "s"(wbase), "v"(adr));
        else if constexpr (k == CNT) asm volatile("s_add_u32 %0, %0, %1" : "+s"(wbase) : "s"(tot) : "scc");
        else if constexpr (k < WR0 + 12) {
            constexpr int c = (k - WR0) / 3, r = (k - WR0) % 3;
            if constexpr (r == 0) asm volatile("v_cndmask_b32 %0, %1, %2, %3" : "=v"(sel) : "v"(dump), "v"(adr), "s"(mk[c]));
            if constexpr (r == 1) *(LAS unsigned*)(size_t)sel = tg[c];
            if constexpr (r == 2 && c < 3) asm volatile("v_add_u32 %0, %1, %2" : "=v"(adr) : "v"(adr), "v"(d4[c]));
        }
        else if constexpr (k == BAR2) {
            asm volatile("" ::: "memory");
            if (!CFG_NOBAR) __builtin_amdgcn_s_barrier();
            asm volatile("" ::: "memory");
        }
        else if constexpr (h == 0) {
            if constexpr (k == HD0) *(LAS unsigned char*)(stg + hoff) = (unsigned char)bits;
            if constexpr (k == PAD0) asm volatile("v_add_u32 %0, %1, %2" : "=v"(sel) : "s"(wbase), "v"(padv));
            if constexpr (k == PAD1) asm volatile("v_min_u32 %0, %1, %2" : "=v"(sel) : "v"(sel), "v"(dump));
            if constexpr (k == PAD2) {
                *(LAS unsigned*)(size_t)sel = TAG;
                __builtin_amdgcn_fence(__ATOMIC_ACQ_REL, "wavefront");
            }
            if constexpr (k >= RD0 && k <= RD2) sendB(k - RD0);
            if constexpr (k >= ST0 && k < ST0 + 6) {
                constexpr int ch = (k - ST0) / 2, r = (k - ST0) % 2;
                if constexpr (r == 0) asm volatile("v_cmp_gt_i32 %0, %1, %2" : "=s"(smk) : "s"(CFG_NOSEND ? 0u : wbase - (unsigned)(size_t)stg - ch * 1024), "v"(l16));
                if constexpr (r == 1) {
                    asm volatile("s_mov_b64 exec, %0\n\ts_cbranch_execz 1f\n\tbuffer_store_dwordx4 %1, %2, %3, %4 offen offset:%5 sc0 sc1\n1:\n\ts_mov_b64 exec, -1"
                                 :: "s"(smk), "v"(sd[ch]), "v"(l16), "s"(rsS), "s"(sso), "i"(ch * 1024) : "memory");
                    if constexpr (ch == 2) { wbase = (unsigned)(size_t)stg + HDR * 4; sso += SLOT; }
                }
            }
        }
    };
    auto grab2 = [&](int c, int j) {
        if (CFG_NOGRAB) asm volatile("" : "=&v"(T[j][c]), "=&v"(T[j + 1][c]) : "a"(acc[c][j]), "a"(acc[c][j + 1])); else
        asm volatile("v_accvgpr_read_b32 %0, %2\n\tv_accvgpr_read_b32 %1, %3"
                     : "=&v"(T[j][c]), "=&v"(T[j + 1][c]) : "a"(acc[c][j]), "a"(acc[c][j + 1]));
    };
#pragma unroll
    for (int c = 0; c < 4; ++c) Bl[c] = *(LAS u4*)(wl + c * 1024);
    T[0][1] = T[1][1] = 0.f;
    __amdgpu_buffer_rsrc_t rs1 = tileA(0);
    int kt = 0;
    SCHED();
    __builtin_amdgcn_s_waitcnt(waitcnt_imm(DX - 3, 15));
    SCHED();

#pragma unroll 1
    for (int it = 0; it < NPK; ++it) {
        sfor<2>([&](auto H) {
            constexpr int h = H;
            const __amdgpu_buffer_rsrc_t rs0 = rs1;
            sfor<128>([&](auto GI) {
                constexpr int G = GI, s = G / 4, c = G % 4;
                u4 f;
                if constexpr (in_lds(s)) f = Bl[c]; else f = Br[reg_idx(s)][c];
                if constexpr (CFG_NOMFMA != 0 && (G % (CFG_NOMFMA ? CFG_NOMFMA : 1)) != 0) { if constexpr (!in_lds(s) && reg_idx(s) < SA) asm volatile("" :: "v"(xw[32 * h + s]), "a"(f)); else asm volatile("" :: "v"(xw[32 * h + s]), "v"(f)); }
                else
                acc[c] = __builtin_amdgcn_mfma_f32_16x16x32_f16(xw[32 * h + s], __builtin_bit_cast(h8, f),
                                                                s == 0 ? f4{0.f, 0.f, 0.f, 0.f} : acc[c], 0, 0, 0);
                SCHED();
                if constexpr (c == 0) {
                    constexpr int sn = s - 1 + DX;
                    if constexpr (sn < 32) issueX(rs0, 32 * h + sn); else issueX(rs1, 32 * h + sn);
                }
                if constexpr (G == 4 * (32 - DX) + 1) { asm volatile("" : "+s"(kt)); rs1 = tileA(++kt); }
                if constexpr (s % 4 == 3) Bl[c] = *(LAS u4*)(wl + (lds_idx((s + 3) % 32) * 4 + c) * 1024);
                if constexpr (CFG_NOFIN) { if constexpr (G >= 124) asm volatile("" :: "a"(acc[c])); } else {
                if constexpr (G == 126) grab2(0, 0);
                if constexpr (G == 127) { grab2(0, 2); grab2(1, 0); }
                if constexpr (G == 0) { grab2(1, 2); grab2(2, 0); }
                if constexpr (G == 1) { grab2(2, 2); grab2(3, 0); }
                if constexpr (G == 2) grab2(3, 2);
                if constexpr (op_at(G, 0) >= 0) side(std::integral_constant<int, op_at(G, 0)>{}, H);
                if constexpr (op_at(G, 1) >= 0) side(std::integral_constant<int, op_at(G, 1)>{}, H);
                static_assert(op_at(G, 2) < 0, "two per gap");
                }
                if constexpr (G % 8 == 7) {
                    constexpr int lg = in_lds(s + 1) ? lds_ops_in(G - 8, G, h) : 15;
                    __builtin_amdgcn_s_waitcnt(waitcnt_imm(DX - 3, lg < 15 ? lg : 15));
                }
                SCHED();
            });
        });
#pragma unroll
        for (int z = 0; z < DX - 1; ++z) xw[z] = xw[64 + z];
    }

    const unsigned rslot0 = ((((call & 1) * NR + u) * 32 + rg * 8 + ci) * 4 + w) * WAVE_SLOTS;
    const __amdgpu_buffer_rsrc_t rsR =
        __builtin_amdgcn_make_buffer_rsrc((void*)(ws + WS_RECV + rslot0), 0, WAVE_SLOTS, 0x00020000);
    const __amdgpu_buffer_rsrc_t rsO = __builtin_amdgcn_make_buffer_rsrc((void*)outp, 0, OUT_BYTES, 0x00020000);
    u4 rd[NPK][PK_CH];
    auto fetch = [&](int pkt, int nch) {
#pragma unroll
        for (int ch = 0; ch < nch; ++ch) {
            const int o = ch * 1024 + l * 16;
            rd[pkt][ch] = __builtin_amdgcn_raw_buffer_load_b128(rsR, o < (int)SLOT ? o : (int)WAVE_SLOTS, pkt * SLOT, AUX_SYS);
        }
    };
#pragma unroll
    for (int k = 0; k < NPK - 1; ++k) fetch(k, 2);
    SCHED();
    cnt = (wbase - ((unsigned)(size_t)stg + HDR * 4)) / 4;
    asm volatile("s_nop 7" ::: "memory");
    grab2(1, 2); grab2(2, 0); grab2(2, 2); grab2(3, 0); grab2(3, 2);
#pragma unroll
    for (int j = 0; j < 4; ++j) putred(j);
    LDS_BARRIER();
#pragma unroll
    for (int o = 0; o < 4; ++o) getred(o);
#pragma unroll
    for (int c = 0; c < 4; ++c) addred(c);
#pragma unroll
    for (int c = 0; c < 4; ++c) packA(c);
    packB(1);
    sfor<6>([&](auto K) { pos = scan_step<K>(pos); });
    packC();
#pragma unroll
    for (int c = 0; c < 4; ++c) packD(c);
    sendA();
#pragma unroll
    for (int ch = 0; ch < PK_CH; ++ch) sendB(ch);
    SCHED();
    __builtin_amdgcn_s_waitcnt(waitcnt_imm(0, 15));
    SCHED();
#pragma unroll
    for (int ch = 0; ch < PK_CH; ++ch) sendC(ch, CFG_NOSEND ? 0u : HDR + cnt, NPK - 1);
    SCHED();
    STAMP(7);

    const int voffO = (4 * g + w) * (NLOC * 4) + q * 16;
    const int baseO = (u * MLOC + rg * 128) * (NLOC * 4) + ci * 64 * 4;
    const int rswz = (CFG_MAP == 2 ? p.rank : ci) & CFG_SWZ;
    unsigned n[NPK];
    u4 o[2 * NPK];
    bool lost = false;
    LAS char* const un = smem + w * (SL * 4096);
    auto expand = [&](auto K0, auto NK, auto NCH) {
        constexpr int k0 = K0, nk = NK, nch = NCH;
        unsigned units[nk], head[nk], by[nk], mine[nk], sum[nk], res = 0;
#pragma unroll
        for (int i = 0; i < nk; ++i) {
            units[i] = 0;
#pragma unroll
            for (int ch = 0; ch < nch; ++ch) {
                const u4 d = rd[k0 + i][ch];
                *(LAS u4*)(un + ((k0 + i) * PK_CH + ch) * 1024 + l * 16) = d;
                const u64 full = __ballot((int)(d[0] & d[1] & d[2] & d[3]) < 0);
                units[i] += __builtin_popcountll(full);
                if (ch == 0) head[i] = (unsigned)full & 63u;
            }
        }
        __builtin_amdgcn_fence(__ATOMIC_ACQ_REL, "wavefront");
#pragma unroll
        for (int i = 0; i < nk; ++i) {
            by[i] = *(LAS unsigned char*)(un + (k0 + i) * (PK_CH * 1024) + hoff);
            sum[i] = mine[i] = __builtin_popcount(by[i] & 15u) | (__builtin_popcount(by[i] >> 4) << 16);
        }
        sfor<6>([&](auto S) {
#pragma unroll
            for (int i = 0; i < nk; ++i) sum[i] = scan_step<S>(sum[i]);
        });
#pragma unroll
        for (int i = 0; i < nk; ++i) {
            const unsigned tot = __builtin_amdgcn_readlane(sum[i], 63), ex = sum[i] - mine[i];
            n[k0 + i] = HDR + (tot & 0xffffu) + (tot >> 16);
            LAS char* at[2] = {un + (k0 + i) * (PK_CH * 1024) + HDR * 4 + (ex & 0xffffu) * 4,
                               un + (k0 + i) * (PK_CH * 1024) + (HDR + (tot & 0xffffu)) * 4 + (ex >> 16) * 4};
#pragma unroll
            for (int e = 0; e < 8; ++e) {
                const int on = __builtin_amdgcn_sbfe((int)by[i], e, 1);
                o[2 * (k0 + i) + (e >> 2)][e & 3] = *(LAS unsigned*)at[e >> 2] & (unsigned)on & ~TAG;
                at[e >> 2] += on & 4;
            }
            res |= (((head[i] == 63u) & (units[i] == (n[k0 + i] + 3) / 4)) | (CFG_NOSEND != 0)) ? 1u << i : 0u;
        }
        __builtin_amdgcn_fence(__ATOMIC_ACQ_REL, "wavefront");
        return res;
    };
    using std::integral_constant;
    constexpr integral_constant<int, 0> i0{};
    constexpr integral_constant<int, 1> i1{};
    constexpr integral_constant<int, 2> i2{};
    constexpr integral_constant<int, 3> i3{};
    auto retry = [&](auto k) {
        const u64 t0 = now();
        for (unsigned spins = 1;; ++spins) {
            fetch(k, PK_CH);
            if (expand(k, i1, i3)) break;
            if ((spins & 15u) == 0 &&
                (__hip_atomic_load(tmo, __ATOMIC_RELAXED, __HIP_MEMORY_SCOPE_AGENT) == call || now() - t0 > DEADLINE_TICKS)) {
                __hip_atomic_store(tmo, call, __ATOMIC_RELAXED, __HIP_MEMORY_SCOPE_AGENT);
                lost = true;
                break;
            }
        }
    };
    auto emit = [&](int k) {
#pragma unroll
        for (int hh = 0; hh < 2; ++hh)
            __builtin_amdgcn_raw_buffer_store_b128(lost ? u4{0x7fc00000u, 0x7fc00000u, 0x7fc00000u, 0x7fc00000u} : o[2 * k + hh],
                                                   rsO, voffO, baseO + ((2 * k + hh) ^ rswz) * 16 * (NLOC * 4), AUX_SC1);
#pragma unroll
        for (int ch = 0; ch < 2; ++ch)
            __builtin_amdgcn_raw_buffer_store_b128(u4{0u, 0u, 0u, 0u}, rsR, ch * 1024 + l * 16, k * SLOT, AUX_SYS);
    };
    {
        const unsigned ok = expand(i0, i3, i2);
        SCHED();
        STAMP(8);
        if (ok != 7u) {
            if (!(ok & 1u)) retry(i0);
            if (!(ok & 2u)) retry(i1);
            if (!(ok & 4u)) retry(i2);
        }
    }
    SCHED();
    STAMP(9);
    fetch(3, 2);
    SCHED();
    emit(0); emit(1); emit(2);
    SCHED();
    STAMP(10);
    if (!expand(i3, i1, i2)) retry(i3);
    SCHED();
    STAMP(11);
    emit(3);
#pragma unroll
    for (int k = 0; k < NPK; ++k)
        if (n[k] > 512u && l < (int)(SLOT - 2048) / 16)
            __builtin_amdgcn_raw_buffer_store_b128(u4{0u, 0u, 0u, 0u}, rsR, 2048 + l * 16, k * SLOT, AUX_SYS);
    if (threadIdx.x == 0) __hip_atomic_store((gu32*)(ws + WS_SEQ) + b, call, __ATOMIC_RELAXED, __HIP_MEMORY_SCOPE_AGENT);
}

}

extern "C" void kernel_launch(int rank, int nranks, ncclComm_t comm,
                              hipStream_t stream,
                              void* const* d_in, const int* in_sizes, int n_in,
                              void* d_out, int out_size,
                              const int* in_dtypes, int out_dtype,
                              void* const* d_out_all,
                              void* const* d_ws_all, size_t ws_bytes) {
    if (nranks != NR || (unsigned)rank >= (unsigned)NR || n_in != 2 || ws_bytes < WS_NEED ||
        in_sizes[0] != MLOC * KD || in_sizes[1] != KD * ND || out_size != NR * MLOC * NLOC) {
        fprintf(stderr, "kernel_launch: unexpected problem shape\n");
        return;
    }
    HIP_CHECK(hipFuncSetAttribute((const void*)gemm_a2a, hipFuncAttributeMaxDynamicSharedMemorySize, LDS_BYTES));
    Params p;
    p.x = (const float*)d_in[0];
    p.w = (const float*)d_in[1];
    p.out = (float*)d_out;
    for (int i = 0; i < NR; ++i) p.ws[i] = (char*)d_ws_all[i];
    p.rank = rank;
    p.pad = 0;
    gemm_a2a<<<NWG, NTHR, LDS_BYTES, stream>>>(p);
    HIP_CHECK(hipGetLastError());
}
```

```cpp
#include <hip/hip_runtime.h>
#include <rccl/rccl.h>
#include <cstdio>
#include <utility>

#define HIP_CHECK(expr) do { hipError_t _e = (expr); if (_e != hipSuccess) { \
    fprintf(stderr, "HIP error %s at %s:%d\n", hipGetErrorString(_e), __FILE__, __LINE__); \
    return; } } while (0)

namespace {

constexpr int NR = 8, MLOC = 512, KD = 4096, ND = 4096, NLOC = 512;
constexpr int NWG = 256, NTHR = 256;

#define CFG_D1 2
#define CFG_DX 13
#define CFG_XS 31
#define CFG_MAP 2
#define CFG_G 4
#define CFG_GS 8
#define CFG_DELAY 0
#define CFG_DGRP 0
#define CFG_PHASE 1
#define CFG_NOSEND 0
#define CFG_NOXLOAD 0
#define CFG_NOMFMA 0
#define CFG_NOBAR 0
#define CFG_NOFIN 0
#define CFG_NOBL 0
#define CFG_NOLD 0
#define CFG_SWZ 0
#define CFG_OLO 0
#define CFG_OHI 1000
#define CFG_NOGRAB 0
#define CFG_POLL 0
#define CFG_IDLE 0
#define CFG_RING 0
#define CFG_FULLSEND 0
#define CFG_NORESET 0
#define CFG_PAD 0
#define CFG_PACE 42
#define CFG_STAMP 0
constexpr int D1 = CFG_D1, DX = CFG_DX;
constexpr int SL = 8, SA = 15, SV = 9;
static_assert(SL + SA + SV == 32 && DX >= 4 && DX <= 24, "k-step split");
constexpr int NPK = 4;
constexpr int HDR = 22;
constexpr int PK_WORDS = HDR + 512;
constexpr int PK_CH = 3;
constexpr unsigned SLOT = 2304;
static_assert((PK_WORDS + 3) * 4 <= SLOT && SLOT <= PK_CH * 1024, "slot size");

constexpr unsigned WS_SEQ = 4096;
constexpr unsigned WS_TMO = 1024;
constexpr unsigned WS_XFLAG = 2048;
constexpr unsigned WS_STAMP = 32768;
constexpr unsigned WS_X16 = 65536;
constexpr unsigned X16_BYTES = MLOC * KD * 2;
constexpr unsigned WS_RECV = WS_X16 + X16_BYTES;
constexpr unsigned WAVE_SLOTS = NPK * SLOT;
constexpr unsigned RECV_BYTES = 2 * NR * 32 * 4 * WAVE_SLOTS;
constexpr size_t WS_NEED = (size_t)WS_RECV + RECV_BYTES;
constexpr unsigned OUT_BYTES = (unsigned)NR * MLOC * NLOC * 4;
constexpr unsigned W_BYTES = (unsigned)KD * ND * 4;
constexpr unsigned X_BYTES = (unsigned)MLOC * KD * 4;
constexpr unsigned long long DEADLINE_TICKS = 50000000ull;
constexpr unsigned TAG = 0x80000000u;

constexpr int LDS_W = 4 * SL * 4096;
constexpr int LDS_RED = LDS_W;
constexpr int LDS_STG = LDS_RED + 16384;
constexpr int STG_B = PK_CH * 1024 + 256;
constexpr int LDS_BYTES = LDS_STG + 4 * STG_B;
static_assert(LDS_BYTES <= 163840, "LDS");

constexpr int AUX_SC1 = 16, AUX_SYS = 17;

typedef float f4 __attribute__((ext_vector_type(4)));
typedef unsigned u4 __attribute__((ext_vector_type(4)));
typedef unsigned u2 __attribute__((ext_vector_type(2)));
typedef unsigned long long u64;
typedef _Float16 h8 __attribute__((ext_vector_type(8)));
typedef _Float16 h2 __attribute__((ext_vector_type(2)));
#define GAS __attribute__((address_space(1)))
#define LAS __attribute__((address_space(3)))
typedef GAS unsigned gu32;

struct Params {
    const float* x;
    const float* w;
    float* out;
    char* ws[NR];
    int rank;
    int pad;
};

template <int... I, class F>
__device__ __forceinline__ void sfor_(std::integer_sequence<int, I...>, F&& f) { (f(std::integral_constant<int, I>{}), ...); }
template <int N, class F>
__device__ __forceinline__ void sfor(F&& f) { sfor_(std::make_integer_sequence<int, N>{}, f); }

__device__ __forceinline__ unsigned pk(unsigned a, unsigned b) {
    h2 v = {(_Float16)__builtin_bit_cast(float, a), (_Float16)__builtin_bit_cast(float, b)};
    return __builtin_bit_cast(unsigned, v);
}
__device__ __forceinline__ u64 now() { return __builtin_amdgcn_s_memrealtime(); }
template <int K>
__device__ __forceinline__ unsigned scan_step(unsigned x) {
    constexpr int ctrl = K < 4 ? 0x110 + (1 << K) : (K == 4 ? 0x142 : 0x143);
    constexpr int rows = K < 4 ? 0xf : (K == 4 ? 0xa : 0xc);
    return x + (unsigned)__builtin_amdgcn_update_dpp(0, (int)x, ctrl, rows, 0xf, false);
}
#define SCHED() __builtin_amdgcn_sched_barrier(0)
#define LDS_BARRIER() do { asm volatile("s_waitcnt lgkmcnt(0)" ::: "memory"); __builtin_amdgcn_s_barrier(); \
                           asm volatile("" ::: "memory"); } while (0)
#define STAMP(i) do { } while (0)

constexpr bool in_lds(int s) { return s % 4 == 2; }
constexpr int lds_idx(int s) { return s / 4; }
constexpr int reg_idx(int s) { int n = 0; for (int i = 0; i < s; ++i) n += in_lds(i) ? 0 : 1; return n; }
constexpr int p1_slot(int i) {
    if (i < SL) return 2 + 4 * i;
    int n = i - SL;
    for (int s = 0; s < 32; ++s) if (!in_lds(s)) { if (n == 0) return s; --n; }
    return -1;
}

enum Op { PUT0, PUT1, PUT2, PUT3, BAR1, GET0, GET1, GET2, GET3, ADD0  , CMP0 = ADD0 + 12  , SEL0 = CMP0 + 4  ,
          OWN0 = SEL0 + 4, OWN1, NIB0, NIB1, NIB2, NIB3, SCN0  , TAG0 = SCN0 + 6  , TOT = TAG0 + 4, ADR0, ADR1, CNT,
          WR0  , HD0 = WR0 + 12, PAD0, PAD1, PAD2, RD0, RD1, RD2, ST0  ,
          BAR2 = ST0 + 6, NOPS };
struct Plan { int gap[NOPS]; };
constexpr int base_load(int G) {
    return (G % 4 == 0) + ((G / 4) % 4 == 3) + (G % 8 == 7);
}
constexpr Plan make_plan() {
    Plan p{};
    int used[128] = {};
    int G = 3;
    for (int k = 0; k < NOPS; ++k) {
        int cost = 1, lo = G;
        if (k <= PUT3 || k == BAR1) cost = 2;
        if (k == BAR1) lo = 17;
        if (k >= GET0 && k <= GET3) lo = G + 1;
        if (k == ADD0) lo = p.gap[GET3] + 10;
        if (k == SEL0) lo = p.gap[CMP0 + 3] + 1;
        if (k == SCN0) lo = p.gap[OWN1] + 2;
        if (k > SCN0 && k < SCN0 + 6) lo = p.gap[k - 1] + 2;
        if (k == TAG0) lo = p.gap[SCN0];
        if (k == TOT) lo = p.gap[SCN0 + 5] + 2;
        if (k == RD0) lo = G + 2;
        if (k == ST0) lo = p.gap[RD2] + 5;
        if (k >= ST0 && k < ST0 + 6 && (k - ST0) % 2 == 1) { lo = p.gap[k - 1] + 1; cost = 2; }
        if (k == BAR2) lo = G + 1;
        while (base_load(lo) + used[lo] + cost > 2) ++lo;
        used[lo] += cost;
        p.gap[k] = lo;
        G = lo;
    }
    return p;
}
constexpr Plan PLAN = make_plan();
static_assert(PLAN.gap[BAR2] < 124 && PLAN.gap[TAG0 + 3] <= PLAN.gap[TOT] && PLAN.gap[PUT3] < 12 && PLAN.gap[BAR1] < 28, "plan");
constexpr int op_at(int G, int n) {
    for (int k = 0; k < NOPS; ++k)
        if (PLAN.gap[k] == G && n-- == 0) return k;
    return -1;
}
constexpr int lds_ops_in(int a, int b, int h) {
    int n = 0;
    for (int k = 0; k < NOPS; ++k) {
        const bool lds = k <= PUT3 || (k >= GET0 && k <= GET3) || (k >= WR0 && k < WR0 + 12 && (k - WR0) % 3 == 1) ||
                         (h == 0 && (k == HD0 || k == PAD2 || (k >= RD0 && k <= RD2)));
        n += lds && PLAN.gap[k] > a && PLAN.gap[k] <= b;
    }
    return n;
}
constexpr int waitcnt_imm(int vm, int lgkm) { return (vm & 15) | (7 << 4) | ((lgkm & 15) << 8) | ((vm >> 4) << 14); }

__global__ __launch_bounds__(NTHR, 1) void gemm_a2a(Params p) {
    extern __shared__ __attribute__((aligned(16))) char smem_[];
    LAS char* smem = (LAS char*)smem_;
    const int b = blockIdx.x;
    const int l = threadIdx.x & 63;
    const int w = __builtin_amdgcn_readfirstlane(threadIdx.x >> 6);
    const int rg = (b >> 3) & 3;
    const int ci = CFG_MAP == 2 ? (b & 7) : (b >> 5);
    const int u = CFG_MAP == 2 ? (b >> 5) : (b & 7);
    const int g = l >> 4, q = l & 15;
    char* const ws = p.ws[p.rank];
    char* wsu = p.ws[u];
    float* outp = p.out;
    STAMP(0);
    const u64 ring0 = CFG_RING ? now() : 0;
    if (CFG_IDLE) { const u64 t00 = now(); while (now() - t00 < CFG_IDLE) __builtin_amdgcn_s_sleep(8); }

    const __amdgpu_buffer_rsrc_t rsX = __builtin_amdgcn_make_buffer_rsrc((void*)p.x, 0, X_BYTES, 0x00020000);
    const __amdgpu_buffer_rsrc_t rsW = __builtin_amdgcn_make_buffer_rsrc((void*)p.w, 0, W_BYTES, 0x00020000);
    const __amdgpu_buffer_rsrc_t rsH = __builtin_amdgcn_make_buffer_rsrc((void*)(ws + WS_X16), 0, X16_BYTES, 0x00020000);
    const int swz = (CFG_MAP == 2 ? u : ci) & CFG_SWZ;
    auto tileA = [&](int k) {
        return __builtin_amdgcn_make_buffer_rsrc((void*)(ws + WS_X16 + (rg * 8 + ((k ^ swz) & 7)) * (128 * 1024)), 0,
                                                 k < 8 ? 128 * 1024 : 0, 0x00020000);
    };

    u4 xs[8];
    {
        const int voff = (l >> 3) * (KD * 4) + (l & 7) * 16;
        const int soff = (b >> 3) * 16 * (KD * 4) + ((b & 7) * 512 + w * 128) * 4;
#pragma unroll
        for (int i = 0; i < 8; ++i)
            xs[i] = __builtin_amdgcn_raw_buffer_load_b128(rsX, voff, soff + (i & 1) * 8 * (KD * 4) + (i >> 1) * 128, 0);
    }
    const u64 done = __hip_atomic_load((GAS u64*)(ws + WS_SEQ) + b, __ATOMIC_RELAXED, __HIP_MEMORY_SCOPE_AGENT);
    SCHED();

    const int line0 = (int)(((size_t)p.w >> 7) & 127) + (u * 8 + ci) * 2;
    const bool slowp = (line0 & 6) == 2 || (line0 & 126) == 62;
    const int gsz = (slowp ? CFG_GS : CFG_G);
    const int rot = rg * (gsz / 4);
    const int ph = (CFG_PHASE == 1 ? ((b >> 5) + (b & 7)) & 3 : (CFG_PHASE == 2 ? (b >> 5) & 3 : (CFG_PHASE == 3 ? (b >> 6) & 3 : 0))) * (gsz / 4);
    auto chunk = [&](int i) { const int a = (i - ph) & 31; return (((a & ~(gsz - 1)) | ((a + rot) & (gsz - 1))) + ph) & 31; };
    if (CFG_DELAY) {
        const int dg = CFG_DGRP == 0 ? (slowp ? 0 : 1) : (CFG_DGRP == 1 ? ((u + ci) & 1) : (CFG_DGRP == 2 ? ((u + ci) & 3) : u));
        const u64 t0 = now();
        while (now() - t0 < (u64)(dg * CFG_DELAY)) __builtin_amdgcn_s_sleep(1);
    }

    u4 st[D1][8];
    const int voffW = g * 8 * (ND * 4) + q * 16;
    const int colW = (u * 8 + ci) * 64 * 4 + w * 32 * (ND * 4);
    auto issueW = [&](int i) {
        const int so = chunk(i) * 128 * (ND * 4) + colW;
#pragma unroll
        for (int j = 0; j < 8; ++j)
            st[i % D1][j] = __builtin_amdgcn_raw_buffer_load_b128(rsW, voffW, so + j * (ND * 4), 0);
    };
#pragma unroll
    for (int i = 0; i < D1; ++i) issueW(i);
    SCHED();

    {
        LAS char* sc = smem + LDS_RED + w * 4096;
        const int woff = ((((l & 7) >> 1) * 16 + (l >> 3)) * 16) + (l & 1) * 8;
#pragma unroll
        for (int i = 0; i < 8; ++i) {
            u2 v = {pk(xs[i][0], xs[i][1]), pk(xs[i][2], xs[i][3])};
            *(LAS u2*)(sc + (i >> 1) * 1024 + (i & 1) * 128 + woff) = v;
        }
        __builtin_amdgcn_fence(__ATOMIC_ACQ_REL, "wavefront");
        const int so = ((b >> 3) * 128 + (b & 7) * 16 + w * 4) * 1024;
#pragma unroll
        for (int ks = 0; ks < 4; ++ks) {
            const u4 v = *(LAS u4*)(sc + ks * 1024 + l * 16);
            __builtin_amdgcn_raw_buffer_store_b128(v, rsH, l * 16, so + ks * 1024, AUX_SC1);
        }
    }
    const unsigned call = __builtin_amdgcn_readfirstlane((unsigned)done) + 1u;
    const bool pace = (unsigned)now() - __builtin_amdgcn_readfirstlane((unsigned)(done >> 32)) < 10000u;
    asm volatile("" : "+s"(wsu), "+s"(outp));
    SCHED();
    STAMP(1);

    u4 Br[SA + SV][4];
    h8 xw[64 + DX];
    LAS char* wl = smem + w * (SL * 4096) + l * 16;
    int xo[32];
    sfor<32>([&](auto I) { xo[p1_slot(I)] = (chunk(I) * 4 + w) * 1024; });
    auto issueX = [&](__amdgpu_buffer_rsrc_t rs, int z) {
        xw[z] = __builtin_bit_cast(h8, __builtin_amdgcn_raw_buffer_load_b128(rs, CFG_NOXLOAD ? 128 * 1024 : l * 16, xo[z % 32], AUX_SC1));
    };
    gu32* const xflag = (gu32*)(ws + WS_XFLAG);
    gu32* const tmo = (gu32*)(ws + WS_TMO);
    unsigned xf = 0;
    bool bad = false;
    constexpr int SPOLL = 16;
    static_assert(SPOLL > D1 && SPOLL + D1 < CFG_XS, "poll placement");
    sfor<32>([&](auto I) {
        constexpr int i = I, s = p1_slot(i);
#pragma unroll
        for (int c = 0; c < 4; ++c) {
            const u4* r = st[i % D1];
            u4 f = {pk(r[0][c], r[1][c]), pk(r[2][c], r[3][c]), pk(r[4][c], r[5][c]), pk(r[6][c], r[7][c])};
            if constexpr (i == 31) f[0] = bad ? 0x7e007e00u : f[0];
            if constexpr (in_lds(s)) *(LAS u4*)(wl + (lds_idx(s) * 4 + c) * 1024) = f;
            else if constexpr (reg_idx(s) < SA) { u4 a; asm("; to acc" : "=a"(a) : "0"(f)); Br[reg_idx(s)][c] = a; }
            else Br[reg_idx(s)][c] = f;
        }
        SCHED();
        if constexpr (i == D1) {
            asm volatile("s_waitcnt vmcnt(%0)" :: "i"(8 * (D1 - 1)) : "memory");
            __builtin_amdgcn_s_barrier();
            if (threadIdx.x == 0) __hip_atomic_store(xflag + b, call, __ATOMIC_RELAXED, __HIP_MEMORY_SCOPE_AGENT);
        }
        if constexpr (i == SPOLL + D1) {
            if (!__all(xf >= call)) {
                const u64 t0 = now();
                while (true) {
                    xf = __hip_atomic_load(xflag + rg * 64 + l, __ATOMIC_RELAXED, __HIP_MEMORY_SCOPE_AGENT);
                    if (__all(xf >= call)) break;
                    if (__hip_atomic_load(tmo, __ATOMIC_RELAXED, __HIP_MEMORY_SCOPE_AGENT) == call || now() - t0 > DEADLINE_TICKS) {
                        __hip_atomic_store(tmo, call, __ATOMIC_RELAXED, __HIP_MEMORY_SCOPE_AGENT);
                        bad = true;
                        break;
                    }
                    __builtin_amdgcn_s_sleep(2);
                }
            }
            __builtin_amdgcn_fence(__ATOMIC_ACQUIRE, "wavefront");
        }
        if constexpr (i + D1 < 32) issueW(i + D1);
        if constexpr (i == SPOLL) xf = __hip_atomic_load(xflag + rg * 64 + l, __ATOMIC_RELAXED, __HIP_MEMORY_SCOPE_AGENT);
        if constexpr (i >= CFG_XS) {
#pragma unroll
            for (int z = 0; z < DX - 1; ++z)
                if (z * (32 - CFG_XS) / (DX - 1) == i - CFG_XS) { issueX(tileA(0), z); SCHED(); }
        }
        SCHED();
    });
    STAMP(2);
    const u64 ring1 = CFG_RING ? now() : 0;

    const unsigned slot0 = ((((call & 1) * NR + p.rank) * 32 + rg * 8 + ci) * 4 + w) * WAVE_SLOTS;
    const __amdgpu_buffer_rsrc_t rsS =
        __builtin_amdgcn_make_buffer_rsrc((void*)(wsu + WS_RECV + slot0), 0, WAVE_SLOTS, 0x00020000);
    LAS char* const redw = smem + LDS_RED + w * 4096 + l * 16;
    LAS char* const redr = smem + LDS_RED + w * 1024 + l * 16;
    LAS char* const stg = smem + LDS_STG + w * STG_B;

    f4 acc[4], T[4], v;
    u4 Bl[4], sd[PK_CH];
    bool nz[4];
    unsigned cnt = 0, bits = 0, nib = 0, pos = 0, own = 0;
    const int hoff = (l / 3) * 4 + l % 3;
    *(LAS unsigned*)(stg + (l < HDR ? l : PK_CH * 256 + l) * 4) = TAG;
#pragma unroll
    for (int c = 0; c < 4; ++c) acc[c] = f4{0.f, 0.f, 0.f, 0.f};
#pragma unroll
    for (int j = 0; j < 4; ++j) T[j][0] = 0.f;

    auto grab = [&](int c) {
#pragma unroll
        for (int j = 0; j < 4; ++j) { float t; asm("; to vgpr" : "=v"(t) : "0"(acc[c][j])); T[j][c] = t; }
    };
    auto putred = [&](int j) { *(LAS f4*)(redw + j * 1024) = T[j]; };
    auto getred = [&](int o) { T[o] = *(LAS f4*)(redr + o * 4096); };
    auto addred = [&](int c) {
        float a = T[0][c] + T[1][c], e = T[2][c] + T[3][c];
        asm("" : "+v"(a));
        v[c] = a + e;
    };
    auto packA = [&](int c) {
        nz[c] = !(v[c] <= 0.f);
        nib = (c == 0 ? 0u : nib) | (nz[c] ? 1u << c : 0u);
    };
    auto packB = [&](int hh) {
        own = __builtin_popcount(nib);
        pos = own;
        bits = hh == 0 ? nib : bits | (nib << 4);
    };
    auto packC = [&]() {
        const unsigned tot = __builtin_amdgcn_readlane(pos, 63);
        pos = pos - own + (HDR + cnt);
        cnt += tot;
    };
    auto packD = [&](int c) {
        const float vc = v[c];
        *(LAS unsigned*)(stg + (nz[c] ? pos : PK_CH * 256 + l) * 4) = __float_as_uint(vc) | TAG;
        pos += nz[c] ? 1u : 0u;
    };
    auto sendA = [&]() {
        *(LAS unsigned char*)(stg + hoff) = (unsigned char)bits;
        *(LAS unsigned*)(stg + (l < 3 ? HDR + cnt + l : PK_CH * 256 + l) * 4) = TAG;
        __builtin_amdgcn_fence(__ATOMIC_ACQ_REL, "wavefront");
    };
    auto sendB = [&](int ch) { sd[ch] = *(LAS u4*)(stg + ch * 1024 + l * 16); };
    auto sendC = [&](int ch, unsigned n, int pkt) {
        if ((unsigned)(ch * 256 + l * 4) < n) __builtin_amdgcn_raw_buffer_store_b128(sd[ch], rsS, ch * 1024 + l * 16, pkt * SLOT, AUX_SYS);
    };

    const unsigned l16 = l * 16, pacei = __builtin_amdgcn_readfirstlane(pace ? 1u : 0u);

    int sso = -(int)SLOT;
    float t0[4] = {}, t1[4] = {};
    unsigned d4[4] = {}, tg[4] = {}, ownb = 0, nb0 = 0, nb1 = 0, adr = 0, sel = 0x7fff0000u, tot = 0;
    u64 mk[4] = {}, smk = 0;
    const unsigned dump = (unsigned)(size_t)(stg + (PK_CH * 256 + l) * 4);
    unsigned wbase = (unsigned)(size_t)stg;
    const unsigned padv = l < 3 ? l * 4u : 0x10000u;
    auto side = [&](auto KK, auto HH) {
        constexpr int k = KK, h = HH;
        if constexpr (k < CFG_OLO || k >= CFG_OHI) { }
        else if constexpr (k <= PUT3) putred(k - PUT0);
        else if constexpr (k == BAR1) {
            asm volatile("s_waitcnt lgkmcnt(4)" ::: "memory");
            if (!CFG_NOBAR) __builtin_amdgcn_s_barrier();
            asm volatile("" ::: "memory");
        }
        else if constexpr (k <= GET3) getred(k - GET0);
        else if constexpr (k < ADD0 + 4) asm volatile("v_add_f32 %0, %1, %2" : "=v"(t0[k - ADD0]) : "v"(T[0][k - ADD0]), "v"(T[1][k - ADD0]));
        else if constexpr (k < ADD0 + 8) asm volatile("v_add_f32 %0, %1, %2" : "=v"(t1[k - ADD0 - 4]) : "v"(T[2][k - ADD0 - 4]), "v"(T[3][k - ADD0 - 4]));
        else if constexpr (k < ADD0 + 12) asm volatile("v_add_f32 %0, %1, %2" : "=v"(t0[k - ADD0 - 8]) : "v"(t0[k - ADD0 - 8]), "v"(t1[k - ADD0 - 8]));
        else if constexpr (k < CMP0 + 4) asm volatile("v_cmp_nge_f32 %0, 0, %1" : "=s"(mk[k - CMP0]) : "v"(t0[k - CMP0]));
        else if constexpr (k < SEL0 + 4) asm volatile("v_cndmask_b32 %0, 0, 4, %1" : "=v"(d4[k - SEL0]) : "s"(mk[k - SEL0]));
        else if constexpr (k == OWN0) asm volatile("v_add3_u32 %0, %1, %2, %3" : "=v"(ownb) : "v"(d4[0]), "v"(d4[1]), "v"(d4[2]));
        else if constexpr (k == OWN1) asm volatile("v_add_u32 %0, %1, %2" : "=v"(ownb) : "v"(ownb), "v"(d4[3]));
        else if constexpr (k == NIB0) asm volatile("v_lshl_add_u32 %0, %1, 1, %2" : "=v"(nb0) : "v"(d4[1]), "v"(d4[0]));
        else if constexpr (k == NIB1) asm volatile("v_lshl_add_u32 %0, %1, 1, %2" : "=v"(nb1) : "v"(d4[3]), "v"(d4[2]));
        else if constexpr (k == NIB2) asm volatile("v_lshl_add_u32 %0, %1, 2, %2" : "=v"(nb0) : "v"(nb1), "v"(nb0));
        else if constexpr (k == NIB3) {
            if constexpr (h == 1) asm volatile("v_lshrrev_b32 %0, 2, %1" : "=v"(bits) : "v"(nb0));
            else asm volatile("v_lshl_or_b32 %0, %1, 2, %2" : "=v"(bits) : "v"(nb0), "v"(bits));
        }
        else if constexpr (k == SCN0) asm volatile("v_add_u32_dpp %0, %1, %1 row_shr:1 row_mask:0xf bank_mask:0xf bound_ctrl:1" : "=&v"(adr) : "v"(ownb));
        else if constexpr (k == SCN0 + 1) asm volatile("v_add_u32_dpp %0, %0, %0 row_shr:2 row_mask:0xf bank_mask:0xf bound_ctrl:1" : "+v"(adr));
        else if constexpr (k == SCN0 + 2) asm volatile("v_add_u32_dpp %0, %0, %0 row_shr:4 row_mask:0xf bank_mask:0xf bound_ctrl:1" : "+v"(adr));
        else if constexpr (k == SCN0 + 3) asm volatile("v_add_u32_dpp %0, %0, %0 row_shr:8 row_mask:0xf bank_mask:0xf bound_ctrl:1" : "+v"(adr));
        else if constexpr (k == SCN0 + 4) asm volatile("v_add_u32_dpp %0, %0, %0 row_bcast:15 row_mask:0xa bank_mask:0xf" : "+v"(adr));
        else if constexpr (k == SCN0 + 5) asm volatile("v_add_u32_dpp %0, %0, %0 row_bcast:31 row_mask:0xc bank_mask:0xf" : "+v"(adr));
        else if constexpr (k < TAG0 + 4) asm volatile("v_or_b32 %0, 0x80000000, %1" : "=v"(tg[k - TAG0]) : "v"(t0[k - TAG0]));
        else if constexpr (k == TOT) asm volatile("v_readlane_b32 %0, %1, 63" : "=s"(tot) : "v"(adr));
        else if constexpr (k == ADR0) asm volatile("v_sub_u32 %0, %1, %2" : "=v"(adr) : "v"(adr), "v"(ownb));
        else if constexpr (k == ADR1) asm volatile("v_add_u32 %0, %1, %2" : "=v"(adr) : "s"(wbase), "v"(adr));
        else if constexpr (k == CNT) asm volatile("s_add_u32 %0, %0, %1" : "+s"(wbase) : "s"(tot) : "scc");
        else if constexpr (k < WR0 + 12) {
            constexpr int c = (k - WR0) / 3, r = (k - WR0) % 3;
            if constexpr (r == 0) asm volatile("v_cndmask_b32 %0, %1, %2, %3" : "=v"(sel) : "v"(dump), "v"(adr), "s"(mk[c]));
            if constexpr (r == 1) *(LAS unsigned*)(size_t)sel = tg[c];
            if constexpr (r == 2 && c < 3) asm volatile("v_add_u32 %0, %1, %2" : "=v"(adr) : "v"(adr), "v"(d4[c]));
        }
        else if constexpr (k == BAR2) {
            asm volatile("" ::: "memory");
            if (!CFG_NOBAR) __builtin_amdgcn_s_barrier();
            asm volatile("s_cmp_eq_u32 %0, 0\n\ts_cbranch_scc1 1f\n\ts_sleep %1\n1:" :: "s"(pacei), "i"(CFG_PACE) : "memory", "scc");
        }
        else if constexpr (h == 0) {
            if constexpr (k == HD0) *(LAS unsigned char*)(stg + hoff) = (unsigned char)bits;
            if constexpr (k == PAD0) asm volatile("v_add_u32 %0, %1, %2" : "=v"(sel) : "s"(wbase), "v"(padv));
            if constexpr (k == PAD1) asm volatile("v_min_u32 %0, %1, %2" : "=v"(sel) : "v"(sel), "v"(dump));
            if constexpr (k == PAD2) {
                *(LAS unsigned*)(size_t)sel = TAG;
                __builtin_amdgcn_fence(__ATOMIC_ACQ_REL, "wavefront");
            }
            if constexpr (k >= RD0 && k <= RD2) sendB(k - RD0);
            if constexpr (k >= ST0 && k < ST0 + 6) {
                constexpr int ch = (k - ST0) / 2, r = (k - ST0) % 2;
                if constexpr (r == 0) asm volatile("v_cmp_gt_i32 %0, %1, %2" : "=s"(smk) : "s"(CFG_NOSEND ? 0u : wbase - (unsigned)(size_t)stg - ch * 1024), "v"(l16));
                if constexpr (r == 1) {
                    asm volatile("s_mov_b64 exec, %0\n\ts_cbranch_execz 1f\n\tbuffer_store_dwordx4 %1, %2, %3, %4 offen offset:%5 sc0 sc1\n1:\n\ts_mov_b64 exec, -1"
                                 :: "s"(smk), "v"(sd[ch]), "v"(l16), "s"(rsS), "s"(sso), "i"(ch * 1024) : "memory");
                    if constexpr (ch == 2) { wbase = (unsigned)(size_t)stg + HDR * 4; sso += SLOT; }
                }
            }
        }
    };
    auto grab2 = [&](int c, int j) {
        if (CFG_NOGRAB) asm volatile("" : "=&v"(T[j][c]), "=&v"(T[j + 1][c]) : "a"(acc[c][j]), "a"(acc[c][j + 1])); else
        asm volatile("v_accvgpr_read_b32 %0, %2\n\tv_accvgpr_read_b32 %1, %3"
                     : "=&v"(T[j][c]), "=&v"(T[j + 1][c]) : "a"(acc[c][j]), "a"(acc[c][j + 1]));
    };
#pragma unroll
    for (int c = 0; c < 4; ++c) Bl[c] = *(LAS u4*)(wl + c * 1024);
    T[0][1] = T[1][1] = 0.f;
    __amdgpu_buffer_rsrc_t rs1 = tileA(0);
    int kt = 0;
    SCHED();
    __builtin_amdgcn_s_waitcnt(waitcnt_imm(DX - 3, 15));
    SCHED();

#pragma unroll 1
    for (int it = 0; it < NPK; ++it) {
        sfor<2>([&](auto H) {
            constexpr int h = H;
            const __amdgpu_buffer_rsrc_t rs0 = rs1;
            sfor<128>([&](auto GI) {
                constexpr int G = GI, s = G / 4, c = G % 4;
                u4 f;
                if constexpr (in_lds(s)) f = Bl[c]; else f = Br[reg_idx(s)][c];
                if constexpr (CFG_NOMFMA != 0 && (G % (CFG_NOMFMA ? CFG_NOMFMA : 1)) != 0) { if constexpr (!in_lds(s) && reg_idx(s) < SA) asm volatile("" :: "v"(xw[32 * h + s]), "a"(f)); else asm volatile("" :: "v"(xw[32 * h + s]), "v"(f)); }
                else
                acc[c] = __builtin_amdgcn_mfma_f32_16x16x32_f16(xw[32 * h + s], __builtin_bit_cast(h8, f),
                                                                s == 0 ? f4{0.f, 0.f, 0.f, 0.f} : acc[c], 0, 0, 0);
                SCHED();
                if constexpr (c == 0) {
                    constexpr int sn = s - 1 + DX;
                    if constexpr (sn < 32) issueX(rs0, 32 * h + sn); else issueX(rs1, 32 * h + sn);
                }
                if constexpr (G == 4 * (32 - DX) + 1) { asm volatile("" : "+s"(kt)); rs1 = tileA(++kt); }
                if constexpr (s % 4 == 3) Bl[c] = *(LAS u4*)(wl + (lds_idx((s + 3) % 32) * 4 + c) * 1024);
                if constexpr (CFG_NOFIN) { if constexpr (G >= 124) asm volatile("" :: "a"(acc[c])); } else {
                if constexpr (G == 126) grab2(0, 0);
                if constexpr (G == 127) { grab2(0, 2); grab2(1, 0); }
                if constexpr (G == 0) { grab2(1, 2); grab2(2, 0); }
                if constexpr (G == 1) { grab2(2, 2); grab2(3, 0); }
                if constexpr (G == 2) grab2(3, 2);
                if constexpr (op_at(G, 0) >= 0) side(std::integral_constant<int, op_at(G, 0)>{}, H);
                if constexpr (op_at(G, 1) >= 0) side(std::integral_constant<int, op_at(G, 1)>{}, H);
                static_assert(op_at(G, 2) < 0, "two per gap");
                }
                if constexpr (CFG_PAD > 0 && c == 2) { for (int z = 0; z < CFG_PAD; ++z) asm volatile("s_nop 7"); }
                if constexpr (G % 8 == 7) {
                    constexpr int lg = in_lds(s + 1) ? lds_ops_in(G - 8, G, h) : 15;
                    __builtin_amdgcn_s_waitcnt(waitcnt_imm(DX - 3, lg < 15 ? lg : 15));
                }
                SCHED();
            });
        });
#pragma unroll
        for (int z = 0; z < DX - 1; ++z) xw[z] = xw[64 + z];
    }

    const unsigned rslot0 = ((((call & 1) * NR + u) * 32 + rg * 8 + ci) * 4 + w) * WAVE_SLOTS;
    const __amdgpu_buffer_rsrc_t rsR =
        __builtin_amdgcn_make_buffer_rsrc((void*)(ws + WS_RECV + rslot0), 0, WAVE_SLOTS, 0x00020000);
    const __amdgpu_buffer_rsrc_t rsO = __builtin_amdgcn_make_buffer_rsrc((void*)outp, 0, OUT_BYTES, 0x00020000);
    u4 rd[NPK][PK_CH];
    auto fetch = [&](int pkt, int nch) {
#pragma unroll
        for (int ch = 0; ch < nch; ++ch) {
            const int o = ch * 1024 + l * 16;
            rd[pkt][ch] = __builtin_amdgcn_raw_buffer_load_b128(rsR, o < (int)SLOT ? o : (int)WAVE_SLOTS, pkt * SLOT, AUX_SYS);
        }
    };
#pragma unroll
    for (int k = 0; k < NPK - 1; ++k) fetch(k, 2);
    SCHED();
    cnt = (wbase - ((unsigned)(size_t)stg + HDR * 4)) / 4;
    asm volatile("s_nop 7" ::: "memory");
    grab2(1, 2); grab2(2, 0); grab2(2, 2); grab2(3, 0); grab2(3, 2);
#pragma unroll
    for (int j = 0; j < 4; ++j) putred(j);
    LDS_BARRIER();
#pragma unroll
    for (int o = 0; o < 4; ++o) getred(o);
#pragma unroll
    for (int c = 0; c < 4; ++c) addred(c);
#pragma unroll
    for (int c = 0; c < 4; ++c) packA(c);
    packB(1);
    sfor<6>([&](auto K) { pos = scan_step<K>(pos); });
    packC();
#pragma unroll
    for (int c = 0; c < 4; ++c) packD(c);
    sendA();
#pragma unroll
    for (int ch = 0; ch < PK_CH; ++ch) sendB(ch);
    SCHED();
    __builtin_amdgcn_s_waitcnt(waitcnt_imm(0, 15));
    SCHED();
#pragma unroll
    for (int ch = 0; ch < PK_CH; ++ch) sendC(ch, CFG_NOSEND ? 0u : HDR + cnt, NPK - 1);
    SCHED();
    STAMP(7);
    const u64 ring2 = CFG_RING ? now() : 0;

    const int voffO = (4 * g + w) * (NLOC * 4) + q * 16;
    const int baseO = (u * MLOC + rg * 128) * (NLOC * 4) + ci * 64 * 4;
    const int rswz = (CFG_MAP == 2 ? p.rank : ci) & CFG_SWZ;
    unsigned n[NPK];
    u4 o[2 * NPK];
    bool lost = false;
    LAS char* const un = smem + w * (SL * 4096);
    auto expand = [&](auto K0, auto NK, auto NCH) {
        constexpr int k0 = K0, nk = NK, nch = NCH;
        unsigned units[nk], head[nk], by[nk], mine[nk], sum[nk], res = 0;
#pragma unroll
        for (int i = 0; i < nk; ++i) {
            units[i] = 0;
#pragma unroll
            for (int ch = 0; ch < nch; ++ch) {
                const u4 d = rd[k0 + i][ch];
                *(LAS u4*)(un + ((k0 + i) * PK_CH + ch) * 1024 + l * 16) = d;
                const u64 full = __ballot((int)(d[0] & d[1] & d[2] & d[3]) < 0);
                units[i] += __builtin_popcountll(full);
                if (ch == 0) head[i] = (unsigned)full & 63u;
            }
        }
        __builtin_amdgcn_fence(__ATOMIC_ACQ_REL, "wavefront");
#pragma unroll
        for (int i = 0; i < nk; ++i) {
            by[i] = *(LAS unsigned char*)(un + (k0 + i) * (PK_CH * 1024) + hoff);
            sum[i] = mine[i] = __builtin_popcount(by[i] & 15u) | (__builtin_popcount(by[i] >> 4) << 16);
        }
        sfor<6>([&](auto S) {
#pragma unroll
            for (int i = 0; i < nk; ++i) sum[i] = scan_step<S>(sum[i]);
        });
#pragma unroll
        for (int i = 0; i < nk; ++i) {
            const unsigned tot = __builtin_amdgcn_readlane(sum[i], 63), ex = sum[i] - mine[i];
            n[k0 + i] = HDR + (tot & 0xffffu) + (tot >> 16);
            LAS char* at[2] = {un + (k0 + i) * (PK_CH * 1024) + HDR * 4 + (ex & 0xffffu) * 4,
                               un + (k0 + i) * (PK_CH * 1024) + (HDR + (tot & 0xffffu)) * 4 + (ex >> 16) * 4};
#pragma unroll
            for (int e = 0; e < 8; ++e) {
                const int on = __builtin_amdgcn_sbfe((int)by[i], e, 1);
                o[2 * (k0 + i) + (e >> 2)][e & 3] = *(LAS unsigned*)at[e >> 2] & (unsigned)on & ~TAG;
                at[e >> 2] += on & 4;
            }
            res |= (((head[i] == 63u) & (units[i] == (n[k0 + i] + 3) / 4)) | (CFG_NOSEND != 0)) ? 1u << i : 0u;
        }
        __builtin_amdgcn_fence(__ATOMIC_ACQ_REL, "wavefront");
        return res;
    };
    using std::integral_constant;
    constexpr integral_constant<int, 0> i0{};
    constexpr integral_constant<int, 1> i1{};
    constexpr integral_constant<int, 2> i2{};
    constexpr integral_constant<int, 3> i3{};
    auto retry = [&](auto k) {
        const u64 t0 = now();
        for (unsigned spins = 1;; ++spins) {
            if (CFG_POLL) {
                for (int z = 0; z < 100000; ++z) {
                    const unsigned hw = __builtin_amdgcn_raw_buffer_load_b32(rsR, 0, k * SLOT, AUX_SYS);
                    if ((int)__builtin_amdgcn_readfirstlane(hw) < 0) break;
                    __builtin_amdgcn_s_sleep(CFG_POLL);
                }
            }
            fetch(k, PK_CH);
            if (expand(k, i1, i3)) break;
            if ((spins & 15u) == 0 &&
                (__hip_atomic_load(tmo, __ATOMIC_RELAXED, __HIP_MEMORY_SCOPE_AGENT) == call || now() - t0 > DEADLINE_TICKS)) {
                __hip_atomic_store(tmo, call, __ATOMIC_RELAXED, __HIP_MEMORY_SCOPE_AGENT);
                lost = true;
                break;
            }
        }
    };
    auto emit = [&](int k) {
#pragma unroll
        for (int hh = 0; hh < 2; ++hh)
            __builtin_amdgcn_raw_buffer_store_b128(lost ? u4{0x7fc00000u, 0x7fc00000u, 0x7fc00000u, 0x7fc00000u} : o[2 * k + hh],
                                                   rsO, voffO, baseO + ((2 * k + hh) ^ rswz) * 16 * (NLOC * 4), AUX_SC1);
#pragma unroll
        for (int ch = 0; ch < (CFG_NORESET ? 0 : 2); ++ch)
            __builtin_amdgcn_raw_buffer_store_b128(u4{0u, 0u, 0u, 0u}, rsR, ch * 1024 + l * 16, k * SLOT, AUX_SYS);
    };
    {
        const unsigned ok = expand(i0, i3, i2);
        SCHED();
        STAMP(8);
        if (ok != 7u) {
            if (!(ok & 1u)) retry(i0);
            if (!(ok & 2u)) retry(i1);
            if (!(ok & 4u)) retry(i2);
        }
    }
    SCHED();
    STAMP(9);
    fetch(3, 2);
    SCHED();
    emit(0); emit(1); emit(2);
    SCHED();
    STAMP(10);
    if (!expand(i3, i1, i2)) retry(i3);
    SCHED();
    STAMP(11);
    emit(3);
#pragma unroll
    for (int k = 0; k < NPK; ++k)
        if (n[k] > 512u && l < (int)(SLOT - 2048) / 16)
            __builtin_amdgcn_raw_buffer_store_b128(u4{0u, 0u, 0u, 0u}, rsR, 2048 + l * 16, k * SLOT, AUX_SYS);
    if (CFG_RING && b == 0 && threadIdx.x == 0) {
        u4 r = {(unsigned)ring0, (unsigned)(ring1 - ring0), (unsigned)(ring2 - ring0), (unsigned)(now() - ring0)};
        *(u4*)(ws + (32u << 20) + (call & 8191u) * 16) = r;
    }
    if (CFG_FULLSEND) {
        const __amdgpu_buffer_rsrc_t rsD = __builtin_amdgcn_make_buffer_rsrc((void*)(wsu + (40u << 20) + ((p.rank * 32 + rg * 8 + ci) * 4 + w) * 4096), 0, 4096, 0x00020000);
#pragma unroll
        for (int z = 0; z < CFG_FULLSEND; ++z) __builtin_amdgcn_raw_buffer_store_b128(o[z], rsD, l * 16, z * 1024, AUX_SYS);
    }
    if (threadIdx.x == 0)
        __hip_atomic_store((GAS u64*)(ws + WS_SEQ) + b, call | (now() << 32), __ATOMIC_RELAXED, __HIP_MEMORY_SCOPE_AGENT);
}

}

extern "C" void kernel_launch(int rank, int nranks, ncclComm_t comm,
                              hipStream_t stream,
                              void* const* d_in, const int* in_sizes, int n_in,
                              void* d_out, int out_size,
                              const int* in_dtypes, int out_dtype,
                              void* const* d_out_all,
                              void* const* d_ws_all, size_t ws_bytes) {
    if (nranks != NR || (unsigned)rank >= (unsigned)NR || n_in != 2 || ws_bytes < WS_NEED ||
        in_sizes[0] != MLOC * KD || in_sizes[1] != KD * ND || out_size != NR * MLOC * NLOC) {
        fprintf(stderr, "kernel_launch: unexpected problem shape\n");
        return;
    }
    HIP_CHECK(hipFuncSetAttribute((const void*)gemm_a2a, hipFuncAttributeMaxDynamicSharedMemorySize, LDS_BYTES));
    Params p;
    p.x = (const float*)d_in[0];
    p.w = (const float*)d_in[1];
    p.out = (float*)d_out;
    for (int i = 0; i < NR; ++i) p.ws[i] = (char*)d_ws_all[i];
    p.rank = rank;
    p.pad = 0;
    gemm_a2a<<<NWG, NTHR, LDS_BYTES, stream>>>(p);
    HIP_CHECK(hipGetLastError());
}
```
